```python
import numpy as np

kernel_name = "gpu_gemm_a2a_m8192_k8192_n8192_f32_none"


ABSMAX_RTOL = 0.001

COMPUTE_DTYPE = "float32"
EPILOGUE = "none"
IN_QUANT = None
OUT_QUANT = None
IN_DTYPES = {"x": "float32", "w_mat": "float32"}
WORLD_SIZE = 8

def setup_inputs():

    x = np.random.randn(8192, 8192).astype(np.float32)
    w_mat = (np.random.randn(8192, 8192) / np.sqrt(8192)).astype(np.float32)
    return {"x": x, "w_mat": w_mat}

def _epilogue(y):
    return y

def reference(x, w_mat):
    y = (x @ w_mat).astype(np.float32)
    return _epilogue(y)
```

```cpp
#include <hip/hip_runtime.h>
#include <rccl/rccl.h>
#include <cstdio>

#define HIP_CHECK(expr) do { hipError_t _e = (expr); if (_e != hipSuccess) { \
    fprintf(stderr, "HIP error %s at %s:%d\n", hipGetErrorString(_e), __FILE__, __LINE__); \
    return; } } while (0)

namespace {

typedef float f4 __attribute__((ext_vector_type(4)));
typedef _Float16 h8 __attribute__((ext_vector_type(8)));
typedef _Float16 h2 __attribute__((ext_vector_type(2)));
typedef unsigned u4 __attribute__((ext_vector_type(4)));
typedef __amdgpu_buffer_rsrc_t rsrc_t;
#define LAS __attribute__((address_space(3)))
#define GAS __attribute__((address_space(1)))

constexpr int NRANK = 8, ML = 1024, KD = 8192, ND = 8192;
#define PLAN 1
constexpr int NRND = 5; constexpr unsigned LS_PACK = 0x33222;
constexpr int NWG = 256, NSTEP = 128;
constexpr size_t WS_XT = 1u << 20, XT_BYTES = (size_t)ML * KD * 2;
constexpr size_t WS_PART = WS_XT + XT_BYTES, IMG_BYTES = 131072, ROUND_BYTES = NWG * IMG_BYTES, PART_BYTES = NRND * ROUND_BYTES;
constexpr size_t WS_NEED = WS_PART + PART_BYTES + (1u << 20);
constexpr int C_TICKET = 0, C_TMO = 32, C_RDONE = 64, C_FLAG = 8 * 32, C_CNT = 16 * 32;
constexpr unsigned long long DEADLINE = 100000000ull;
constexpr int SLOT = 16384, L_SEND = 2 * SLOT, L_CALL = L_SEND + 114688, L_PROG = L_CALL + 16, L_PF = L_CALL + 64, LDS_BYTES = L_PF + 256;
#define PF_DIST 5
#define S_TAU 22
#define S_GMIN 700
#define S_WIN 5
#define S_CATCH 2
#define K_ROT 1

struct Params {
  const float* x; const float* W;
  char* out[NRANK]; char* ws[NRANK];
  int rank, phase;
};

#define SB0() __builtin_amdgcn_sched_barrier(0)
#define SGB(m, n) __builtin_amdgcn_sched_group_barrier(m, n, 0)
#define ASF4(r) __builtin_bit_cast(f4, r)
#define ASU4(r) __builtin_bit_cast(u4, r)
#define ASH8(r) __builtin_bit_cast(h8, r)
#define BAR() do { __builtin_amdgcn_s_barrier(); SB0(); } while (0)
#define BARL() do { asm volatile("s_waitcnt lgkmcnt(0)" ::: "memory"); __builtin_amdgcn_s_barrier(); SB0(); } while (0)
#define WAIT_VM0() asm volatile("s_waitcnt vmcnt(0)" ::: "memory")
#define ST128(v, r, vo, so, aux) do { const u4 d_ = (v); __builtin_amdgcn_raw_buffer_store_b128(d_, r, vo, so, aux); asm volatile("s_nop 1" :: "v"(d_) : "memory"); } while (0)
#define RSRC(p, n) __builtin_amdgcn_make_buffer_rsrc((void*)(p), 0, (n), 0x00020000)

__device__ __forceinline__ unsigned pk(float a, float b) { h2 v = {(_Float16)a, (_Float16)b}; return __builtin_bit_cast(unsigned, v); }
__device__ __forceinline__ int vm_now() {
  unsigned v; asm volatile("s_getreg_b32 %0, hwreg(HW_REG_IB_STS)" : "=s"(v) :: "memory");
  return (int)((v & 15u) | ((v >> 18) & 0x30u));
}
__host__ __device__ constexpr int ls_of(int r) { return (int)((LS_PACK >> (4 * r)) & 15u); }
__host__ __device__ constexpr int g_beg(int r) { return r == 0 ? 0 : g_beg(r - 1) + (128 >> ls_of(r - 1)); }
__device__ __forceinline__ int g_beg_rt(int r) {
  int v = 0;
#pragma unroll
  for (int i = 0; i < NRND; ++i) v = r > i ? g_beg(i + 1) : v;
  return v;
}
__device__ __forceinline__ int split_of(int r, int xcd) { return xcd >> (3 - ls_of(r)); }
__device__ __forceinline__ int tile_idx(int r, int xcd, int jq) { return (g_beg_rt(r) >> 1) + (xcd & ((8 >> ls_of(r)) - 1)) * 8 + jq; }
__device__ __forceinline__ int n_tile(int r, int rank, int xcd, int jq) {
  const int i = tile_idx(r, xcd, jq);
  return i >= 56 ? rank * 8 + (i - 56) : ((rank + 1 + i % 7) & 7) * 8 + i / 7;
}
__device__ __forceinline__ int k_beg(int r, int xcd) { return ((split_of(r, xcd) + K_ROT * r) & ((1 << ls_of(r)) - 1)) * (128 >> ls_of(r)); }
__device__ __forceinline__ unsigned w_base(int r, int rank, int xcd, int jq) {
  if (r >= NRND) return 0x20000000u;
  return (unsigned)((k_beg(r, xcd) - g_beg_rt(r)) * (64 * ND * 4) + n_tile(r, rank, xcd, jq) * 512);
}
__device__ __forceinline__ unsigned x_base(int r, int xcd) {
  if (r >= NRND) return 0x20000000u;
  return (unsigned)((k_beg(r, xcd) - g_beg_rt(r)) * 8192);
}
__device__ __forceinline__ int rot_of(int r, int xcd) { return r >= NRND ? 0 : split_of(r, xcd) * (8 >> ls_of(r)); }
__device__ __forceinline__ unsigned ld_agent(const GAS unsigned* p) { return __hip_atomic_load(p, __ATOMIC_RELAXED, __HIP_MEMORY_SCOPE_AGENT); }

#define STAMP(i) do {} while (0)

__global__ void __launch_bounds__(512, 2) fused_gemm_a2a(const Params P) {
  extern __shared__ __attribute__((aligned(16))) char smem_[];
  LAS char* smem = (LAS char*)smem_;
  const int tid = threadIdx.x, lane = tid & 63, wid = __builtin_amdgcn_readfirstlane(tid >> 6);
  const int rank = P.rank;
  char* const ws = P.ws[rank];

  if (P.phase == 0) {
    if (blockIdx.x < 256 && wid == 0) STAMP(19);
    const int u = blockIdx.x * 8 + wid;
    const int mb = u >> 6, kq = u & 63;
    const float* src = P.x + (size_t)(mb * 16 + (lane & 15)) * KD + kq * 128 + (lane >> 4) * 8;
    u4* xt = (u4*)(ws + WS_XT);
    f4 v[8];
#pragma unroll
    for (int i = 0; i < 4; ++i) { v[2 * i] = *(const f4*)(src + i * 32); v[2 * i + 1] = *(const f4*)(src + i * 32 + 4); }
#pragma unroll
    for (int i = 0; i < 4; ++i) {
      u4 o = {pk(v[2 * i][0], v[2 * i][1]), pk(v[2 * i][2], v[2 * i][3]), pk(v[2 * i + 1][0], v[2 * i + 1][1]), pk(v[2 * i + 1][2], v[2 * i + 1][3])};
      xt[((size_t)((mb >> 2) * 256 + kq * 4 + i) * 4 + (mb & 3)) * 64 + lane] = o;
    }
    return;
  }

  const int xcd = blockIdx.x & 7, q = blockIdx.x >> 3, mt = q & 3, jq = q >> 2;
  GAS unsigned* const ctl = (GAS unsigned*)ws;
  constexpr unsigned WSTEP = 64u * ND * 4u;
  constexpr int NBAR = NSTEP;
  const rsrc_t rW = RSRC(P.W, 0x10000000u);
  const rsrc_t rP = RSRC(ws + WS_PART, (unsigned)PART_BYTES);

  if (wid == 7) {
    unsigned call = 0;
    if (lane == 0) call = __hip_atomic_fetch_add(ctl + C_TICKET, 1u, __ATOMIC_RELAXED, __HIP_MEMORY_SCOPE_AGENT);
    call = __builtin_amdgcn_readfirstlane(call) / NWG + 1;
    if (lane == 0) { *(volatile LAS unsigned*)(smem + L_CALL) = call; *(volatile LAS unsigned*)(smem + L_PROG) = 0; }
    const int pv = (mt * 16 + (lane >> 2)) * (ND * 4) + (lane & 3) * 128;
    unsigned wb[NRND];
#pragma unroll
    for (int r = 0; r < NRND; ++r) wb[r] = w_base(r, rank, xcd, jq);
    const unsigned pfl = (unsigned)(uintptr_t)(smem + L_PF);
#define PREFETCH(g) do { unsigned so_ = wb[0], keep_; \
      _Pragma("unroll") for (int i_ = 1; i_ < NRND; ++i_) so_ = (g) >= g_beg(i_) ? wb[i_] : so_; \
      so_ += (unsigned)(g) * WSTEP; \
      if ((g) < NSTEP && vm_now() < 48) asm volatile("s_mov_b32 %0, m0\n\ts_mov_b32 m0, %1\n\ts_nop 0\n\tbuffer_load_dword %2, %3, %4 offen lds\n\ts_mov_b32 m0, %0" \
        : "=&s"(keep_) : "s"(pfl), "v"(pv), "s"(rW), "s"(so_) : "memory"); } while (0)
#define TICK_S() do { if (nb < NBAR) { PREFETCH(nb + PF_DIST); BAR(); ++nb; } else __builtin_amdgcn_s_sleep(1); } while (0)
    for (int g = 3; g < PF_DIST; ++g) PREFETCH(g);
    BARL();
    int nb = 0;
    unsigned long long tnext = 0;
    for (int r = 0; r < NRND - 1; ++r) {
      const int ti = tile_idx(r, xcd, jq), nt = n_tile(r, rank, xcd, jq), lt = 3 - ls_of(r);
      const unsigned dst = (unsigned)(nt >> 3);
      const rsrc_t rO = RSRC(P.out[dst < NRANK ? dst : 0], dst < NRANK ? 0x2000000u : 0u);
      const unsigned ob = (unsigned)((rank * ML + mt * 256) * 4096 + (nt & 7) * 512 + rot_of(r, xcd) * 64);
      const int srow = lane >> (2 + lt), sc = lane & ((4 << lt) - 1);
      const int ub = g_beg_rt(r);
      const int lb = L_SEND + ub * 1024 + (sc >> 2) * 1024 + ((sc & 3) * 16 + srow) * 16, ov = srow * 4096 + sc * 16;
      const int lsub = 256 >> lt, osub = (16 >> lt) * 4096;
      const int nu = 16 << lt;
      const int i0 = ub >> 1, i1 = g_beg_rt(r + 1) >> 1, here = (i1 + 6 - ti % 7) / 7 - (i0 + 6 - ti % 7) / 7;
      const unsigned gap = (unsigned)max(here * (512 / nu) * S_TAU, S_GMIN);
      for (int u = 0; u < nu; ++u) {
        const int need = ub + (((u >> lt) + 1) << lt);
        while ((int)__builtin_amdgcn_readfirstlane(*(volatile LAS unsigned*)(smem + L_PROG)) < need) TICK_S();
        if (u == 0) STAMP(8 + r);
        { const unsigned long long now = __builtin_amdgcn_s_memrealtime() * 10ull; if (tnext + S_CATCH * gap < now) tnext = now - S_CATCH * gap; }
        const u4 v = *(const LAS u4*)(smem + lb + ((u >> lt) << (10 + lt)) + (u & ((1 << lt) - 1)) * lsub);
        while (__builtin_amdgcn_s_memrealtime() * 10ull < tnext || vm_now() >= S_WIN) TICK_S();
        ST128(v, rO, ov, ob + (unsigned)((u >> lt) * 16 * 4096 + (u & ((1 << lt) - 1)) * osub), 17);
        tnext += gap;
      }
      STAMP(12 + r);
    }
    while (nb < NBAR) TICK_S();
    WAIT_VM0();
    STAMP(16);
    unsigned old = 0;
    if (lane == 0) old = __hip_atomic_fetch_add(ctl + C_RDONE, 1u, __ATOMIC_RELEASE, __HIP_MEMORY_SCOPE_AGENT);
    old = __builtin_amdgcn_readfirstlane(old);
    if ((old & (NWG - 1)) == NWG - 1) {
      __builtin_amdgcn_fence(__ATOMIC_ACQUIRE, "agent");
      __builtin_amdgcn_fence(__ATOMIC_RELEASE, "");
      WAIT_VM0();
      const bool peer = lane < NRANK && lane != rank;
      char* pw = P.ws[0];
#pragma unroll
      for (int p = 1; p < NRANK; ++p) pw = lane == p ? P.ws[p] : pw;
      if (peer) __hip_atomic_store((GAS unsigned*)pw + C_FLAG + rank * 32, call, __ATOMIC_RELAXED, __HIP_MEMORY_SCOPE_SYSTEM);
      STAMP(17);
      const unsigned long long t1 = __builtin_amdgcn_s_memrealtime();
      for (;;) {
        unsigned f = call;
        if (peer) f = __hip_atomic_load(ctl + C_FLAG + lane * 32, __ATOMIC_RELAXED, __HIP_MEMORY_SCOPE_SYSTEM);
        if (__all(f >= call)) break;
        if (ld_agent(ctl + C_TMO) == call || __builtin_amdgcn_s_memrealtime() - t1 > DEADLINE) {
          __hip_atomic_store(ctl + C_TMO, call, __ATOMIC_RELAXED, __HIP_MEMORY_SCOPE_AGENT);
          if (lane == 0) ((float*)P.out[rank])[(size_t)rank * ML * 1024] = __builtin_nanf("");
          break;
        }
        __builtin_amdgcn_s_sleep(2);
      }
      __builtin_amdgcn_fence(__ATOMIC_ACQUIRE, "");
      STAMP(18);
    }
    return;
  }

  if (wid == 6) {
    BAR();
    const unsigned call = __builtin_amdgcn_readfirstlane(*(volatile LAS unsigned*)(smem + L_CALL));
    int nb = 0; bool bad = false;
    unsigned long long t0 = 0;
#define TICK_R() do { if (nb < NBAR) { BAR(); ++nb; } else { \
      if (t0 == 0) t0 = __builtin_amdgcn_s_memrealtime(); \
      else if (__builtin_amdgcn_s_memrealtime() - t0 > DEADLINE) bad = true; \
      __builtin_amdgcn_s_sleep(1); } } while (0)
#define REDUCE(r) do { constexpr int S_ = 1 << ls_of(r), UB_ = 16 / S_, SL_ = 131072 / S_; \
      while (nb < g_beg((r) + 1) + 5) TICK_R(); \
      for (;;) { \
        const unsigned c_ = ld_agent(ctl + C_CNT + ((r) * NWG + blockIdx.x) * 32 + (lane & 3) * 8); \
        do { TICK_R(); } while (vm_now() != 0); \
        if (__all(c_ >= call * (S_ - 1))) break; \
        if (nb >= NBAR && ld_agent(ctl + C_TMO) == call) bad = true; \
        if (bad) { __hip_atomic_store(ctl + C_TMO, call, __ATOMIC_RELAXED, __HIP_MEMORY_SCOPE_AGENT); break; } \
      } \
      STAMP(25 + (r)); \
      const unsigned pb_ = (unsigned)(r) * (unsigned)ROUND_BYTES + blockIdx.x * (unsigned)IMG_BYTES; \
      u4 ld_[UB_][S_ - 1]; \
      for (int b = 0; b <= 8; ++b) { \
        if (b > 0) { \
          do { TICK_R(); } while (vm_now() != 0); \
          _Pragma("unroll") for (int i = 0; i < UB_; ++i) { \
            LAS f4* p_ = (LAS f4*)(smem + L_SEND + (g_beg(r) + (b - 1) * UB_ + i) * 1024 + lane * 16); \
            f4 a_ = *p_; \
            _Pragma("unroll") for (int e = 0; e < S_ - 1; ++e) a_ += ASF4(ld_[i][e]); \
            if (bad) a_ = (f4){__builtin_nanf(""), __builtin_nanf(""), __builtin_nanf(""), __builtin_nanf("")}; \
            *p_ = a_; } \
          asm volatile("" ::: "memory"); \
          *(volatile LAS unsigned*)(smem + L_PROG) = g_beg(r) + b * UB_; SB0(); \
        } \
        if (b < 8) { _Pragma("unroll") for (int i = 0; i < UB_; ++i) _Pragma("unroll") for (int e = 0; e < S_ - 1; ++e) \
          ld_[i][e] = __builtin_amdgcn_raw_buffer_load_b128(rP, lane * 16, pb_ + e * SL_ + (b * UB_ + i) * 1024, 16); SB0(); } \
      } } while (0)
    REDUCE(0); REDUCE(1); REDUCE(2);
    REDUCE(3);
    while (nb < NBAR) { BAR(); ++nb; }
    return;
  }

  if (wid >= 4) {
    const int c = lane & 31, kg0 = (wid - 4) * 4 + (lane >> 5);
    const int wv = (kg0 * 8 * ND + c * 4) * 4;
    const int wrest = (kg0 >> 2) * 1024 + (kg0 & 3) * 256 + (c & 3) * 16;
    u4 wr[2][2][8];
#define WOFF(rot) ((((c >> 2) - (rot)) & 7) * 2048 + wrest + ((((c >> 2) - (rot)) & 1) << 6))
#define WLOADU(set, u, so) _Pragma("unroll") for (int j = 0; j < 8; ++j) \
      wr[set][u][j] = __builtin_amdgcn_raw_buffer_load_b128(rW, wv, (so) + ((u) * 16 + j) * (ND * 4), 0)
#define WCVT(set, u, i) (u4){pk(ASF4(wr[set][u][0])[i], ASF4(wr[set][u][1])[i]), pk(ASF4(wr[set][u][2])[i], ASF4(wr[set][u][3])[i]), \
                             pk(ASF4(wr[set][u][4])[i], ASF4(wr[set][u][5])[i]), pk(ASF4(wr[set][u][6])[i], ASF4(wr[set][u][7])[i])}
#define WWRITEU(set, u, buf, wo) _Pragma("unroll") for (int i = 0; i < 4; ++i) \
      *(LAS u4*)(smem + (buf) * SLOT + ((wo) ^ ((i & 1) << 6)) + (i >> 1) * 128 + (u) * 512) = WCVT(set, u, i)
    unsigned cur = w_base(0, rank, xcd, jq);
    int wcur = WOFF(rot_of(0, xcd));
    WLOADU(0, 0, cur); WLOADU(0, 1, cur); SB0(); WLOADU(1, 0, cur + WSTEP); WLOADU(1, 1, cur + WSTEP); SB0();
    WWRITEU(0, 0, 0, wcur); WWRITEU(0, 1, 0, wcur); WLOADU(0, 0, cur + 2 * WSTEP); WLOADU(0, 1, cur + 2 * WSTEP); SB0();
    BARL();
    for (int r = 0; r < NRND; ++r) {
      const int ge = g_beg_rt(r + 1);
      const unsigned nxt = w_base(r + 1, rank, xcd, jq);
      const int wnxt = WOFF(rot_of(r + 1, xcd));
      for (int g = g_beg_rt(r); g < ge; g += 2) {
#pragma unroll
        for (int z = 0; z < 2; ++z) {
          const unsigned wso = (g + z + 3 < ge ? cur : nxt) + (unsigned)(g + z + 3) * WSTEP;
          const int wo = g + z + 1 < ge ? wcur : wnxt;
          WWRITEU((z + 1) & 1, 0, (z + 1) & 1, wo); SB0(); WLOADU((z + 1) & 1, 0, wso); SB0();
          WWRITEU((z + 1) & 1, 1, (z + 1) & 1, wo); SB0(); WLOADU((z + 1) & 1, 1, wso); SB0();
          BARL();
        }
      }
      cur = nxt; wcur = wnxt;
    }
    return;
  }

  __builtin_amdgcn_s_setprio(1);
  STAMP(0);
  const rsrc_t rX = RSRC(ws + WS_XT, (unsigned)XT_BYTES);
  const int rcp = (lane >> 2) & 3;
  const int rd0 = (lane >> 4) * 256 + (rcp | (((lane & 1) ^ 0) << 2) | ((lane & 2) << 2)) * 16;
  const int rd1 = (lane >> 4) * 256 + (rcp | (((lane & 1) ^ 1) << 2) | ((lane & 2) << 2)) * 16;
  const int xv = lane * 16;
  const unsigned xs0 = (unsigned)(mt * 4 + wid) * (256u * 4096u);
  f4 acc[4][8];
  u4 xf[4][4], wf[8];
#define XLOAD(j, so) _Pragma("unroll") for (int m = 0; m < 4; ++m) xf[j][m] = __builtin_amdgcn_raw_buffer_load_b128(rX, xv + m * 1024, (so), 0)
#define FRD(t, buf, kk) wf[t] = *(const LAS u4*)(smem + (buf) * SLOT + (((t) & 1) ? rd1 : rd0) + ((t) * 2 + (kk)) * 1024)
#define MM(t, j) _Pragma("unroll") for (int m = 0; m < 4; ++m) acc[m][t] = __builtin_amdgcn_mfma_f32_16x16x32_f16(ASH8(wf[t]), ASH8(xf[j][m]), acc[m][t], 0, 0, 0)
#define STEP(buf, so) do { \
    _Pragma("unroll") for (int t = 0; t < 8; ++t) { MM(t, (buf) * 2); FRD(t, buf, 1); SGB(0x8, 4); SGB(0x100, 1); } \
    XLOAD((buf) * 2, (so)); SGB(0x20, 4); SB0(); \
    MM(0, (buf) * 2 + 1); MM(1, (buf) * 2 + 1); SB0(); \
    BARL(); \
    FRD(0, (buf) ^ 1, 0); FRD(1, (buf) ^ 1, 0); SGB(0x100, 2); \
    _Pragma("unroll") for (int t = 2; t < 8; ++t) { MM(t, (buf) * 2 + 1); FRD(t, (buf) ^ 1, 0); SGB(0x8, 4); SGB(0x100, 1); } \
    XLOAD((buf) * 2 + 1, (so) + 4096); SGB(0x20, 4); SB0(); } while (0)
#define PARTNER(r, b) (q * 8 + ((((split_of(r, xcd) + (b)) & ((1 << ls_of(r)) - 1)) << (3 - ls_of(r))) | (xcd & ((8 >> ls_of(r)) - 1))))
#define PUT_PARTIALS(r) do { constexpr int S_ = 1 << ls_of(r), OW_ = 8 / S_; \
    _Pragma("unroll") for (int b = 1; b < S_; ++b) { \
      const unsigned o_ = (unsigned)(r) * (unsigned)ROUND_BYTES + wid * (4096 * OW_) + (unsigned)PARTNER(r, b) * (unsigned)IMG_BYTES + (b - 1) * (16384 * OW_); \
      _Pragma("unroll") for (int m = 0; m < 4; ++m) _Pragma("unroll") for (int t = 0; t < OW_; ++t) \
        ST128(ASU4(acc[m][b * OW_ + t]), rP, xv + t * 1024, o_ + m * (OW_ * 1024), 16); } } while (0)
#define SIGNAL(r) do { if (lane >= 1 && lane < (1 << ls_of(r))) \
    __hip_atomic_fetch_add(ctl + C_CNT + ((r) * NWG + PARTNER(r, lane)) * 32 + wid * 8, 1u, __ATOMIC_RELAXED, __HIP_MEMORY_SCOPE_AGENT); } while (0)
#define ZERO_ACC() _Pragma("unroll") for (int m = 0; m < 4; ++m) _Pragma("unroll") for (int t = 0; t < 8; ++t) acc[m][t] = (f4){0, 0, 0, 0}
#define PAIR(g) do { const unsigned so = ((g) + 2 < ge ? cur : nxt) + (unsigned)((g) + 2) * 8192u; STEP(0, so); STEP(1, so + 8192u); } while (0)
#define ROUND(r) do { ZERO_ACC(); \
    const int ge = g_beg((r) + 1); const unsigned nxt = xs0 + x_base((r) + 1, xcd); \
    _Pragma("unroll 1") for (int g = g_beg(r); g < ge; g += 2) { PAIR(g); \
      if ((r) > 0 && g == g_beg(r) + 2) {     \
        asm volatile("s_waitcnt vmcnt(16)" ::: "memory"); SIGNAL((r) > 0 ? (r) - 1 : 0); SB0(); if (wid == 0) STAMP(2 + (r)); } } \
    cur = nxt; if (wid == 0) STAMP(20 + (r)); } while (0)
  unsigned cur = xs0 + x_base(0, xcd);
  XLOAD(0, cur); XLOAD(1, cur + 4096); XLOAD(2, cur + 8192); XLOAD(3, cur + 12288); SB0();
  BAR();
  const unsigned call = __builtin_amdgcn_readfirstlane(*(volatile LAS unsigned*)(smem + L_CALL));
  bool bad = false;
#pragma unroll
  for (int t = 0; t < 8; ++t) FRD(t, 0, 0);
  SB0();
  STAMP(1);
#define FULL_ROUND(r) do { constexpr int OW_ = 8 >> ls_of(r); ROUND(r); PUT_PARTIALS(r); \
    _Pragma("unroll") for (int m = 0; m < 4; ++m) _Pragma("unroll") for (int t = 0; t < OW_; ++t) \
      *(LAS f4*)(smem + L_SEND + g_beg(r) * 1024 + wid * (4096 * OW_) + xv + (m * OW_ + t) * 1024) = acc[m][t]; \
    SB0(); } while (0)
  FULL_ROUND(0); FULL_ROUND(1); FULL_ROUND(2);
  FULL_ROUND(3);
  ROUND(NRND - 1);
  PUT_PARTIALS(NRND - 1);
  WAIT_VM0();
  SIGNAL(NRND - 1);
  if (wid == 0) STAMP(7);
  { const GAS unsigned* cnt = ctl + C_CNT + ((NRND - 1) * NWG + blockIdx.x) * 32 + wid * 8;
    const unsigned long long t0 = __builtin_amdgcn_s_memrealtime();
    while (__builtin_amdgcn_readfirstlane(ld_agent(cnt)) < call * 7u) {
      if (ld_agent(ctl + C_TMO) == call || __builtin_amdgcn_s_memrealtime() - t0 > DEADLINE) {
        __hip_atomic_store(ctl + C_TMO, call, __ATOMIC_RELAXED, __HIP_MEMORY_SCOPE_AGENT); bad = true; break; }
      __builtin_amdgcn_s_sleep(1); } }
  if (wid == 0) STAMP(29);
#pragma unroll
  for (int b = 1; b < 8; ++b)
#pragma unroll
    for (int m = 0; m < 4; ++m)
      acc[m][b] = ASF4(__builtin_amdgcn_raw_buffer_load_b128(rP, xv + m * 1024, (unsigned)(NRND - 1) * (unsigned)ROUND_BYTES + blockIdx.x * (unsigned)IMG_BYTES + wid * 4096 + (b - 1) * 16384, 16));
#pragma unroll
  for (int m = 0; m < 4; ++m) {
#pragma unroll
    for (int b = 1; b < 8; ++b) acc[m][0] += acc[m][b];
    if (bad) acc[m][0] = (f4){__builtin_nanf(""), __builtin_nanf(""), __builtin_nanf(""), __builtin_nanf("")};
  }
  {
    const rsrc_t rO = RSRC(P.out[rank], 0x2000000u);
    const unsigned ob = (unsigned)((rank * ML + mt * 256 + wid * 64) * 4096 + jq * 512 + xcd * 64);
    const int ov = (lane & 15) * 4096 + (lane >> 4) * 16;
#pragma unroll
    for (int m = 0; m < 4; ++m) ST128(ASU4(acc[m][0]), rO, ov, ob + m * (16 * 4096), 0);
    if (wid == 0) STAMP(2);
  }
}

}

extern "C" void kernel_launch(int rank, int nranks, ncclComm_t comm,
                              hipStream_t stream,
                              void* const* d_in, const int* in_sizes, int n_in,
                              void* d_out, int out_size,
                              const int* in_dtypes, int out_dtype,
                              void* const* d_out_all,
                              void* const* d_ws_all, size_t ws_bytes) {
  if (nranks != NRANK || ws_bytes < WS_NEED || n_in < 2 || out_size != ND * ML) {
    fprintf(stderr, "unsupported configuration\n");
    return;
  }
  HIP_CHECK(hipFuncSetAttribute((const void*)fused_gemm_a2a, hipFuncAttributeMaxDynamicSharedMemorySize, LDS_BYTES));
  Params P;
  P.x = (const float*)d_in[0]; P.W = (const float*)d_in[1];
  for (int p = 0; p < NRANK; ++p) { P.out[p] = (char*)d_out_all[p]; P.ws[p] = (char*)d_ws_all[p]; }
  P.rank = rank;
  P.phase = 0;
  hipLaunchKernelGGL(fused_gemm_a2a, dim3(512), dim3(512), 0, stream, P);
  P.phase = 1;
  hipLaunchKernelGGL(fused_gemm_a2a, dim3(NWG), dim3(512), LDS_BYTES, stream, P);
  HIP_CHECK(hipGetLastError());
}
```

```cpp
#include <hip/hip_runtime.h>
#include <rccl/rccl.h>
#include <cstdio>

#define HIP_CHECK(expr) do { hipError_t _e = (expr); if (_e != hipSuccess) { \
    fprintf(stderr, "HIP error %s at %s:%d\n", hipGetErrorString(_e), __FILE__, __LINE__); \
    return; } } while (0)

namespace {

typedef float f4 __attribute__((ext_vector_type(4)));
typedef _Float16 h8 __attribute__((ext_vector_type(8)));
typedef _Float16 h2 __attribute__((ext_vector_type(2)));
typedef unsigned u4 __attribute__((ext_vector_type(4)));
typedef __amdgpu_buffer_rsrc_t rsrc_t;
#define LAS __attribute__((address_space(3)))
#define GAS __attribute__((address_space(1)))

constexpr int NRANK = 8, ML = 1024, KD = 8192, ND = 8192;
#define PLAN 2
constexpr int NRND = 3; constexpr unsigned LS_PACK = 0x221;
constexpr int NWG = 256, NSTEP = 128;
constexpr size_t WS_XT = 1u << 20, XT_BYTES = (size_t)ML * KD * 2;
constexpr size_t WS_PART = WS_XT + XT_BYTES, IMG_BYTES = 131072, ROUND_BYTES = NWG * IMG_BYTES, PART_BYTES = NRND * ROUND_BYTES;
constexpr size_t WS_NEED = WS_PART + PART_BYTES + (1u << 20);
constexpr int C_TICKET = 0, C_TMO = 32, C_RDONE = 64, C_FLAG = 8 * 32, C_CNT = 16 * 32, C_XRDY = C_CNT + 5 * 256 * 32;
constexpr unsigned long long DEADLINE = 100000000ull;
constexpr int SLOT = 16384, L_SEND = 2 * SLOT, L_CALL = L_SEND + 114688, L_PROG = L_CALL + 16, L_PF = L_CALL + 64, LDS_BYTES = L_PF + 256;
#define PF_DIST 5
#define S_RATE 26
#define H_PRIO 2
#define R_UB 8
#define SIG_AT 0
#define S_WIN 5
#define S_CATCH 2
#define K_REV 0
#define X_AUX 16

struct Params {
  const float* x; const float* W;
  char* out[NRANK]; char* ws[NRANK];
  int rank;
};

#define SB0() __builtin_amdgcn_sched_barrier(0)
#define SGB(m, n) __builtin_amdgcn_sched_group_barrier(m, n, 0)
#define ASF4(r) __builtin_bit_cast(f4, r)
#define ASU4(r) __builtin_bit_cast(u4, r)
#define ASH8(r) __builtin_bit_cast(h8, r)
#define BAR() do { __builtin_amdgcn_s_barrier(); SB0(); } while (0)
#define BARL() do { asm volatile("s_waitcnt lgkmcnt(0)" ::: "memory"); __builtin_amdgcn_s_barrier(); SB0(); } while (0)
#define WAIT_VM0() asm volatile("s_waitcnt vmcnt(0)" ::: "memory")
#define ST128(v, r, vo, so, aux) do { const u4 d_ = (v); __builtin_amdgcn_raw_buffer_store_b128(d_, r, vo, so, aux); asm volatile("s_nop 1" :: "v"(d_) : "memory"); } while (0)
#define RSRC(p, n) __builtin_amdgcn_make_buffer_rsrc((void*)(p), 0, (n), 0x00020000)

__device__ __forceinline__ unsigned pk(float a, float b) { h2 v = {(_Float16)a, (_Float16)b}; return __builtin_bit_cast(unsigned, v); }
__device__ __forceinline__ int lane_now() {
  int l; asm volatile("v_mbcnt_lo_u32_b32 %0, -1, 0\n\tv_mbcnt_hi_u32_b32 %0, -1, %0" : "=v"(l)); return l;
}
__device__ __forceinline__ int vm_now() {
  unsigned v; asm volatile("s_getreg_b32 %0, hwreg(HW_REG_IB_STS)" : "=s"(v) :: "memory");
  return (int)((v & 15u) | ((v >> 18) & 0x30u));
}
__host__ __device__ constexpr int ls_of(int r) { return (int)((LS_PACK >> (4 * r)) & 15u); }
__host__ __device__ constexpr int g_beg(int r) { return r == 0 ? 0 : g_beg(r - 1) + (128 >> ls_of(r - 1)); }
__device__ __forceinline__ int g_beg_rt(int r) {
  int v = 0;
#pragma unroll
  for (int i = 0; i < NRND; ++i) v = r > i ? g_beg(i + 1) : v;
  return v;
}
__device__ __forceinline__ int split_of(int r, int xcd) { return xcd >> (3 - ls_of(r)); }
__device__ __forceinline__ int n_tile(int r, int rank, int xcd, int jq) {
  return ((rank + jq) & 7) * 8 + (g_beg_rt(r) >> 4) + (xcd & ((8 >> ls_of(r)) - 1));
}
#define K_ILV 0
__device__ __forceinline__ unsigned k_stride(int r, unsigned unit) {
  const unsigned st = K_ILV ? unit << ls_of(r) : unit;
  return r >= NRND ? 0u : (K_REV ? 0u - st : st);
}
__device__ __forceinline__ unsigned k_base(int r, int xcd, unsigned unit) {
  if (r >= NRND) return 0x20000000u;
  const int k0 = K_ILV ? split_of(r, xcd) - (g_beg_rt(r) << ls_of(r))
                       : ((split_of(r, xcd) + r) & ((1 << ls_of(r)) - 1)) * (128 >> ls_of(r)) - g_beg_rt(r);
  return (unsigned)(K_REV ? 127 - k0 : k0) * unit;
}
__device__ __forceinline__ unsigned w_base(int r, int rank, int xcd, int jq) {
  return k_base(r, xcd, 64u * ND * 4u) + (r >= NRND ? 0u : (unsigned)n_tile(r, rank, xcd, jq) * 512u);
}
__device__ __forceinline__ int rot_of(int r, int xcd) { return r >= NRND ? 0 : split_of(r, xcd) * (8 >> ls_of(r)); }
__device__ __forceinline__ unsigned ld_agent(const GAS unsigned* p) { return __hip_atomic_load(p, __ATOMIC_RELAXED, __HIP_MEMORY_SCOPE_AGENT); }

#define STAMP(i) do {} while (0)

__global__ void __launch_bounds__(512, 2) fused_gemm_a2a(const Params P) {
  extern __shared__ __attribute__((aligned(16))) char smem_[];
  LAS char* smem = (LAS char*)smem_;
  const int tid = threadIdx.x, lane = tid & 63, wid = __builtin_amdgcn_readfirstlane(tid >> 6);
  const int rank = P.rank;
  char* const ws = P.ws[rank];

  if (wid == 6) STAMP(0);
  const int xcd = blockIdx.x & 7, q = blockIdx.x >> 3, mt = q & 3, jq = q >> 2;
  GAS unsigned* const ctl = (GAS unsigned*)ws;
  constexpr unsigned WSTEP = 64u * ND * 4u;
  constexpr int NBAR = NSTEP;
  const rsrc_t rW = RSRC(P.W, 0x10000000u);
  const rsrc_t rP = RSRC(ws + WS_PART, (unsigned)PART_BYTES);

  const rsrc_t rX = RSRC(ws + WS_XT, (unsigned)XT_BYTES);
  {
    const int mb = blockIdx.x & 63, r8 = lane >> 3, c8 = lane & 7;
    f4 v[2][8];
#pragma unroll
    for (int h = 0; h < 2; ++h) {
      const float* src = P.x + (size_t)(mb * 16 + r8) * KD + ((blockIdx.x >> 6) * 16 + wid * 2 + h) * 128 + c8 * 4;
#pragma unroll
      for (int i = 0; i < 8; ++i) v[h][i] = *(const f4*)(src + (i & 1) * 8 * KD + (i >> 1) * 32);
    }
    typedef unsigned u2 __attribute__((ext_vector_type(2)));
    LAS char* tp = smem + L_SEND + wid * 8192;
#pragma unroll
    for (int h = 0; h < 2; ++h)
#pragma unroll
      for (int i = 0; i < 8; ++i)
        *(LAS u2*)(tp + (h * 4 + (i >> 1)) * 1024 + ((c8 >> 1) * 16 + (i & 1) * 8 + r8) * 16 + (c8 & 1) * 8) = (u2){pk(v[h][i][0], v[h][i][1]), pk(v[h][i][2], v[h][i][3])};
    if (wid == 6) STAMP(5);
#pragma unroll
    for (int h = 0; h < 2; ++h)
#pragma unroll
      for (int i = 0; i < 4; ++i)
        ST128(*(const LAS u4*)(tp + (h * 4 + i) * 1024 + lane * 16), rX, lane * 16,
              (unsigned)((((mb >> 2) * 256 + ((blockIdx.x >> 6) * 16 + wid * 2 + h) * 4 + i) * 4 + (mb & 3)) * 1024), 16);
    WAIT_VM0();
    if (wid == 6) STAMP(6);
    BARL();
  }
  if (wid == 6) STAMP(19);

  if (wid >= 6) __builtin_amdgcn_s_setprio(H_PRIO);
  if (wid == 7) {
#define COUNT_X(h) do { if (lane < 8) __hip_atomic_fetch_add(ctl + C_XRDY + ((((blockIdx.x >> 4) & 3) * 2 + (h)) * 8 + lane) * 32, 1u, __ATOMIC_RELAXED, __HIP_MEMORY_SCOPE_AGENT); } while (0)
    COUNT_X(0);
    unsigned ticket = 0;
    if (lane == 0) ticket = __hip_atomic_fetch_add(ctl + C_TICKET, 1u, __ATOMIC_RELAXED, __HIP_MEMORY_SCOPE_AGENT);
    const unsigned call = __builtin_amdgcn_readfirstlane(ticket) / NWG + 1;
    if (lane == 0) { *(volatile LAS unsigned*)(smem + L_CALL) = call; *(volatile LAS unsigned*)(smem + L_PROG) = 0; }
    const int pv = (mt * 16 + (lane >> 2)) * (ND * 4) + (lane & 3) * 128;
    unsigned wb[NRND];
#pragma unroll
    for (int r = 0; r < NRND; ++r) wb[r] = w_base(r, rank, xcd, jq);
    const unsigned pfl = (unsigned)(uintptr_t)(smem + L_PF);
#define PREFETCH(g) do { unsigned so_ = wb[0], st_ = k_stride(0, WSTEP), keep_; \
      _Pragma("unroll") for (int i_ = 1; i_ < NRND; ++i_) { so_ = (g) >= g_beg(i_) ? wb[i_] : so_; st_ = (g) >= g_beg(i_) ? k_stride(i_, WSTEP) : st_; } \
      so_ = __builtin_amdgcn_readfirstlane(so_ + (unsigned)(g) * st_); \
      if ((g) < NSTEP) asm volatile("s_mov_b32 %0, m0\n\ts_mov_b32 m0, %1\n\ts_nop 3\n\tbuffer_load_dword %2, %3, %4 offen lds\n\ts_mov_b32 m0, %0" \
        : "=&s"(keep_) : "s"(pfl), "v"(pv), "s"(rW), "s"(so_) : "memory"); } while (0)
#define TICK_S() do { if (nb < NBAR) { BAR(); ++nb; PREFETCH(nb + PF_DIST); } else __builtin_amdgcn_s_sleep(4); } while (0)
    for (int g = 3; g <= PF_DIST; ++g) PREFETCH(g);
#define WAIT_X(h) do { const unsigned long long t0_ = __builtin_amdgcn_s_memrealtime(); \
      while (__builtin_amdgcn_readfirstlane(ld_agent(ctl + C_XRDY + ((mt * 2 + (h)) * 8 + xcd) * 32)) < call * 64u) { \
        if (ld_agent(ctl + C_TMO) == call || __builtin_amdgcn_s_memrealtime() - t0_ > DEADLINE) { \
          __hip_atomic_store(ctl + C_TMO, call, __ATOMIC_RELAXED, __HIP_MEMORY_SCOPE_AGENT); break; } \
        __builtin_amdgcn_s_sleep(2); } } while (0)
    STAMP(10);
    WAIT_X(0);
    STAMP(11);
    BARL();
    int nb = 0;

    int credit = 0;
    for (int r = 0; r < NRND - 1; ++r) {
      const int nt = n_tile(r, rank, xcd, jq), lt = 3 - ls_of(r);
      const unsigned dst = (unsigned)(nt >> 3);
      const rsrc_t rO = RSRC(P.out[dst < NRANK ? dst : 0], dst < NRANK ? 0x2000000u : 0u);
      const unsigned ob = (unsigned)((rank * ML + mt * 256) * 4096 + (nt & 7) * 512 + rot_of(r, xcd) * 64);
      const int srow = lane >> (2 + lt), sc = lane & ((4 << lt) - 1);
      const int ub = g_beg_rt(r);
      const int lb = L_SEND + ub * 1024 + (sc >> 2) * 1024 + ((sc & 3) * 16 + srow) * 16, ov = srow * 4096 + sc * 16;
      const int lsub = 256 >> lt, osub = (16 >> lt) * 4096, lm = (1 << lt) - 1;
      const int nu = 16 << lt;
      bool first = true;
      for (int u = 0; u < nu;) {
        const unsigned pg = *(volatile LAS unsigned*)(smem + L_PROG);
        const int u1 = u + 1 < nu ? u + 1 : u;
        const u4 v0 = *(const volatile LAS u4*)(smem + lb + ((u >> lt) << (10 + lt)) + (u & lm) * lsub);
        const u4 v1 = *(const volatile LAS u4*)(smem + lb + ((u1 >> lt) << (10 + lt)) + (u1 & lm) * lsub);
        SB0();
        TICK_S();
        credit = min(credit + S_RATE, 40);
        const int have = (int)__builtin_amdgcn_readfirstlane(pg) - ub, room = S_WIN - vm_now();
        if (have > u && credit >= 16 && room > 0) {
          if (first) { STAMP(8 + r); first = false; }
          ST128(v0, rO, ov, ob + (unsigned)((u >> lt) * 16 * 4096 + (u & lm) * osub), 17);
          credit -= 16; ++u;
          if (u < nu && have > u && credit >= 16 && room > 1) {
            ST128(v1, rO, ov, ob + (unsigned)((u >> lt) * 16 * 4096 + (u & lm) * osub), 17);
            credit -= 16; ++u;
          }
        }
      }
      STAMP(12 + r);
    }
    while (nb < NBAR) TICK_S();
    WAIT_VM0();
    BAR();
    STAMP(16);
    unsigned old = 0;
    if (lane == 0) old = __hip_atomic_fetch_add(ctl + C_RDONE, 1u, __ATOMIC_RELEASE, __HIP_MEMORY_SCOPE_AGENT);
    old = __builtin_amdgcn_readfirstlane(old);
    if ((old & (NWG - 1)) == NWG - 1) {
      __builtin_amdgcn_fence(__ATOMIC_ACQUIRE, "agent");
      __builtin_amdgcn_fence(__ATOMIC_RELEASE, "");
      WAIT_VM0();
      const bool peer = lane < NRANK && lane != rank;
      char* pw = P.ws[0];
#pragma unroll
      for (int p = 1; p < NRANK; ++p) pw = lane == p ? P.ws[p] : pw;
      if (peer) __hip_atomic_store((GAS unsigned*)pw + C_FLAG + rank * 32, call, __ATOMIC_RELAXED, __HIP_MEMORY_SCOPE_SYSTEM);
      STAMP(17);
      const unsigned long long t1 = __builtin_amdgcn_s_memrealtime();
      for (;;) {
        unsigned f = call;
        if (peer) f = __hip_atomic_load(ctl + C_FLAG + lane * 32, __ATOMIC_RELAXED, __HIP_MEMORY_SCOPE_SYSTEM);
        if (__all(f >= call)) break;
        if (ld_agent(ctl + C_TMO) == call || __builtin_amdgcn_s_memrealtime() - t1 > DEADLINE) {
          __hip_atomic_store(ctl + C_TMO, call, __ATOMIC_RELAXED, __HIP_MEMORY_SCOPE_AGENT);
          if (lane == 0) ((float*)P.out[rank])[(size_t)rank * ML * 1024] = __builtin_nanf("");
          break;
        }
        __builtin_amdgcn_s_sleep(2);
      }
      __builtin_amdgcn_fence(__ATOMIC_ACQUIRE, "");
      STAMP(18);
    }
    return;
  }

  if (wid == 6) {
    BAR();
    const unsigned call = __builtin_amdgcn_readfirstlane(*(volatile LAS unsigned*)(smem + L_CALL));
    int nb = 0; bool bad = false;
    unsigned long long t0 = 0;
#define TICK_R() do { if (nb < NBAR) { BAR(); ++nb; } else { \
      if (t0 == 0) t0 = __builtin_amdgcn_s_memrealtime(); \
      else if (__builtin_amdgcn_s_memrealtime() - t0 > DEADLINE) bad = true; \
      __builtin_amdgcn_s_sleep(1); } } while (0)
#define REDUCE(r) do { constexpr int S_ = 1 << ls_of(r), OW_ = 8 / S_, UB_ = (R_UB / (S_ - 1) / OW_ > 0 ? R_UB / (S_ - 1) / OW_ : 1) * OW_, NB_ = 128 / S_ / UB_, SL_ = 131072 / S_; \
      while (nb < g_beg((r) + 1) + SIG_AT + 2) TICK_R(); \
      for (;;) { \
        const unsigned c_ = ld_agent(ctl + C_CNT + ((r) * NWG + blockIdx.x) * 32 + (lane & 3) * 8), tm_ = ld_agent(ctl + C_TMO); \
        do { TICK_R(); } while (vm_now() != 0); \
        if (__builtin_amdgcn_readfirstlane(tm_) == call) bad = true; \
        if (__all(c_ >= call * (S_ - 1))) break; \
        if (bad) { __hip_atomic_store(ctl + C_TMO, call, __ATOMIC_RELAXED, __HIP_MEMORY_SCOPE_AGENT); break; } \
      } \
      STAMP(25 + (r)); \
      const unsigned pb_ = (unsigned)(r) * (unsigned)ROUND_BYTES + blockIdx.x * (unsigned)IMG_BYTES; \
      u4 ld_[UB_][S_ - 1]; f4 own_[UB_]; \
      _Pragma("unroll 1") for (int b = 0; b < NB_; ++b) { \
        LAS f4* p_ = (LAS f4*)(smem + L_SEND + (g_beg(r) + b * UB_) * 1024 + lane * 16); \
        _Pragma("unroll") for (int i = 0; i < UB_; ++i) { own_[i] = p_[i * 64]; _Pragma("unroll") for (int e = 0; e < S_ - 1; ++e) \
          ld_[i][e] = __builtin_amdgcn_raw_buffer_load_b128(rP, lane * 16, pb_ + e * SL_ + (b * UB_ + i) * 1024, 16); } \
        SB0(); \
        do { TICK_R(); } while (vm_now() != 0); \
        _Pragma("unroll") for (int i = 0; i < UB_; ++i) { \
          f4 a_ = own_[i]; \
          _Pragma("unroll") for (int e = 0; e < S_ - 1; ++e) a_ += ASF4(ld_[i][e]); \
          if (bad) a_ = (f4){__builtin_nanf(""), __builtin_nanf(""), __builtin_nanf(""), __builtin_nanf("")}; \
          p_[i * 64] = a_; } \
        asm volatile("" ::: "memory"); \
        *(volatile LAS unsigned*)(smem + L_PROG) = g_beg(r) + (b + 1) * UB_; SB0(); \
      } } while (0)
    REDUCE(0);
    if constexpr (NRND > 2) REDUCE(NRND > 2 ? 1 : 0);
    if constexpr (NRND > 3) REDUCE(NRND > 3 ? 2 : 0);
    if constexpr (NRND > 4) REDUCE(NRND > 4 ? 3 : 0);
    while (nb < NBAR) { BAR(); ++nb; }
    return;
  }

  if (wid >= 4) {
    const int c = lane & 31, kg0 = (wid - 4) * 4 + (lane >> 5);
    const int wv = (kg0 * 8 * ND + c * 4) * 4;
    const int wrest = (kg0 >> 2) * 1024 + (kg0 & 3) * 256 + (c & 3) * 16;
    u4 wr[2][2][8];
#define WOFF(rot) ((((c >> 2) - (rot)) & 7) * 2048 + wrest + ((((c >> 2) - (rot)) & 1) << 6))
#define WLOADU(set, u, so) _Pragma("unroll") for (int j = 0; j < 8; ++j) \
      wr[set][u][j] = __builtin_amdgcn_raw_buffer_load_b128(rW, wv, (so) + ((u) * 16 + j) * (ND * 4), 0)
#define WCVT(set, u, i) (u4){pk(ASF4(wr[set][u][0])[i], ASF4(wr[set][u][1])[i]), pk(ASF4(wr[set][u][2])[i], ASF4(wr[set][u][3])[i]), \
                             pk(ASF4(wr[set][u][4])[i], ASF4(wr[set][u][5])[i]), pk(ASF4(wr[set][u][6])[i], ASF4(wr[set][u][7])[i])}
#define WWRITEU(set, u, buf, wo) _Pragma("unroll") for (int i = 0; i < 4; ++i) \
      *(LAS u4*)(smem + (buf) * SLOT + ((wo) ^ ((i & 1) << 6)) + (i >> 1) * 128 + (u) * 512) = WCVT(set, u, i)
    unsigned cur = w_base(0, rank, xcd, jq);
    int wcur = WOFF(rot_of(0, xcd));
    unsigned scur = k_stride(0, WSTEP);
    WLOADU(0, 0, cur); WLOADU(0, 1, cur); SB0(); WLOADU(1, 0, cur + scur); WLOADU(1, 1, cur + scur); SB0();
    WWRITEU(0, 0, 0, wcur); WWRITEU(0, 1, 0, wcur); WLOADU(0, 0, cur + 2 * scur); WLOADU(0, 1, cur + 2 * scur); SB0();
    BARL();
    for (int r = 0; r < NRND; ++r) {
      const int ge = g_beg_rt(r + 1);
      const unsigned nxt = w_base(r + 1, rank, xcd, jq);
      const unsigned snxt = k_stride(r + 1, WSTEP);
      const int wnxt = WOFF(rot_of(r + 1, xcd));
      for (int g = g_beg_rt(r); g < ge; g += 2) {
#pragma unroll
        for (int z = 0; z < 2; ++z) {
          const unsigned wso = g + z + 3 < ge ? cur + (unsigned)(g + z + 3) * scur : nxt + (unsigned)(g + z + 3) * snxt;
          const int wo = g + z + 1 < ge ? wcur : wnxt;
          WWRITEU((z + 1) & 1, 0, (z + 1) & 1, wo); SB0(); WLOADU((z + 1) & 1, 0, wso); SB0();
          WWRITEU((z + 1) & 1, 1, (z + 1) & 1, wo); SB0(); WLOADU((z + 1) & 1, 1, wso); SB0();
          BARL();
        }
      }
      cur = nxt; scur = snxt; wcur = wnxt;
    }
    return;
  }

  __builtin_amdgcn_s_setprio(1);
  const int rcp = (lane >> 2) & 3;
  const int rd0 = (int)(uintptr_t)smem + (lane >> 4) * 256 + (rcp | (((lane & 1) ^ 0) << 2) | ((lane & 2) << 2)) * 16;
  const int rd1 = (int)(uintptr_t)smem + (lane >> 4) * 256 + (rcp | (((lane & 1) ^ 1) << 2) | ((lane & 2) << 2)) * 16;
  const int xv = lane * 16;
  const unsigned xs0 = (unsigned)(mt * 4 + wid) * (256u * 4096u);
  f4 acc[4][8] = {};
  u4 xf[4][4], wf[8] = {};
#define XLOAD(j, so) _Pragma("unroll") for (int m = 0; m < 4; ++m) xf[j][m] = __builtin_amdgcn_raw_buffer_load_b128(rX, xv + m * 1024, (so), X_AUX)
#define FRD(t, buf, kk) asm volatile("ds_read_b128 %0, %1 offset:%2" : "+v"(wf[t]) : "v"(((t) & 1) ? rd1 : rd0), "n"((buf) * SLOT + ((t) * 2 + (kk)) * 1024) : "memory")
#define WAIT_L(n) asm volatile("s_waitcnt lgkmcnt(" #n ")" ::: "memory")
#define MM(t, j, z) _Pragma("unroll") for (int m = 0; m < 4; ++m) { \
    if (z) asm volatile("v_mfma_f32_16x16x32_f16 %0, %1, %2, 0" : "+v"(acc[m][t]) : "v"(wf[t]), "v"(xf[j][m])); \
    else asm volatile("v_mfma_f32_16x16x32_f16 %0, %1, %2, %0" : "+v"(acc[m][t]) : "v"(wf[t]), "v"(xf[j][m])); }
#define STEP(buf, so, z) do { \
    _Pragma("unroll") for (int t = 0; t < 8; ++t) { WAIT_L(7); MM(t, (buf) * 2, z); FRD(t, buf, 1); SB0(); } \
    XLOAD((buf) * 2, (so)); SB0(); \
    WAIT_L(7); MM(0, (buf) * 2 + 1, 0); WAIT_L(6); MM(1, (buf) * 2 + 1, 0); SB0(); \
    BARL(); \
    FRD(0, (buf) ^ 1, 0); FRD(1, (buf) ^ 1, 0); SB0(); \
    _Pragma("unroll") for (int t = 2; t < 8; ++t) { MM(t, (buf) * 2 + 1, 0); FRD(t, (buf) ^ 1, 0); SB0(); } \
    XLOAD((buf) * 2 + 1, (so) + 4096); SB0(); } while (0)
#define PARTNER(r, b) (q * 8 + ((((split_of(r, xcd) + (b)) & ((1 << ls_of(r)) - 1)) << (3 - ls_of(r))) | (xcd & ((8 >> ls_of(r)) - 1))))
#define PUT_PARTIALS(r) do { constexpr int S_ = 1 << ls_of(r), OW_ = 8 / S_; \
    _Pragma("unroll") for (int b = 1; b < S_; ++b) { \
      const unsigned o_ = (unsigned)(r) * (unsigned)ROUND_BYTES + wid * (4096 * OW_) + (unsigned)PARTNER(r, b) * (unsigned)IMG_BYTES + (b - 1) * (16384 * OW_); \
      _Pragma("unroll") for (int m = 0; m < 4; ++m) _Pragma("unroll") for (int t = 0; t < OW_; ++t) \
        ST128(ASU4(acc[m][b * OW_ + t]), rP, xv + t * 1024, o_ + m * (OW_ * 1024), 16); } } while (0)
#define SIGNAL(r) do { const int l_ = lane_now(); if (l_ >= 1 && l_ < (1 << ls_of(r))) \
    __hip_atomic_fetch_add(ctl + C_CNT + ((r) * NWG + PARTNER(r, l_)) * 32 + wid * 8, 1u, __ATOMIC_RELAXED, __HIP_MEMORY_SCOPE_AGENT); } while (0)
#define PAIR(r, g, z) do { const unsigned st = (g) + 2 < ge ? k_stride(r, 8192u) : k_stride((r) + 1, 8192u), so = ((g) + 2 < ge ? cur : nxt) + (unsigned)((g) + 2) * st; \
    STEP(0, so, z); STEP(1, so + st, 0); } while (0)
#define ROUND(r) do { \
    const int ge = g_beg((r) + 1); const unsigned nxt = xs0 + k_base((r) + 1, xcd, 8192u); \
    PAIR(r, g_beg(r), 1); \
    if ((r) > 0) {     \
      asm volatile("s_waitcnt vmcnt(16)" ::: "memory"); SIGNAL((r) > 0 ? (r) - 1 : 0); SB0(); if (wid == 0) STAMP(2 + (r)); } \
    _Pragma("unroll 1") for (int g = g_beg(r) + 2; g < ge; g += 2) PAIR(r, g, 0); \
    asm volatile("s_nop 15" ::: "memory"); SB0();     \
    cur = nxt; if (wid == 0) STAMP(20 + (r)); } while (0)
  unsigned cur = xs0 + k_base(0, xcd, 8192u);
  BAR();
  XLOAD(0, cur); XLOAD(1, cur + 4096); XLOAD(2, cur + k_stride(0, 8192u)); XLOAD(3, cur + k_stride(0, 8192u) + 4096); SB0();
  unsigned call = __builtin_amdgcn_readfirstlane(*(volatile LAS unsigned*)(smem + L_CALL));
  asm volatile("" : "+s"(call)); SB0();
  bool bad = false;
#pragma unroll
  for (int t = 0; t < 8; ++t) FRD(t, 0, 0);
  SB0();
  STAMP(1);
#define FULL_ROUND(r) do { constexpr int OW_ = 8 >> ls_of(r); ROUND(r); PUT_PARTIALS(r); \
    _Pragma("unroll") for (int m = 0; m < 4; ++m) _Pragma("unroll") for (int t = 0; t < OW_; ++t) \
      *(LAS f4*)(smem + L_SEND + g_beg(r) * 1024 + wid * (4096 * OW_) + xv + (m * OW_ + t) * 1024) = acc[m][t]; \
    SB0(); } while (0)
  FULL_ROUND(0);
  if constexpr (NRND > 2) FULL_ROUND(NRND > 2 ? 1 : 0);
  if constexpr (NRND > 3) FULL_ROUND(NRND > 3 ? 2 : 0);
  if constexpr (NRND > 4) FULL_ROUND(NRND > 4 ? 3 : 0);
  constexpr int RL = NRND - 1, SL = 1 << ls_of(RL), OL = 8 / SL;
  ROUND(RL);
  PUT_PARTIALS(RL);
  WAIT_VM0();
  SIGNAL(RL);
  if (wid == 0) STAMP(7);
  { const GAS unsigned* cnt = ctl + C_CNT + (RL * NWG + blockIdx.x) * 32 + wid * 8;
    const unsigned long long t0 = __builtin_amdgcn_s_memrealtime();
    while (__builtin_amdgcn_readfirstlane(ld_agent(cnt)) < call * (SL - 1)) {
      if (ld_agent(ctl + C_TMO) == call || __builtin_amdgcn_s_memrealtime() - t0 > DEADLINE) {
        __hip_atomic_store(ctl + C_TMO, call, __ATOMIC_RELAXED, __HIP_MEMORY_SCOPE_AGENT); bad = true; break; }
      __builtin_amdgcn_s_sleep(1); } }
  if (ld_agent(ctl + C_TMO) == call) bad = true;
  if (wid == 0) STAMP(29);
#pragma unroll
  for (int b = 1; b < SL; ++b)
#pragma unroll
    for (int m = 0; m < 4; ++m)
#pragma unroll
      for (int t = 0; t < OL; ++t)
        acc[m][b * OL + t] = ASF4(__builtin_amdgcn_raw_buffer_load_b128(rP, xv + t * 1024,
            (unsigned)RL * (unsigned)ROUND_BYTES + blockIdx.x * (unsigned)IMG_BYTES + wid * (4096 * OL) + (b - 1) * (16384 * OL) + m * (OL * 1024), 16));
  {
    const int nt = n_tile(RL, rank, xcd, jq);
    const unsigned dst = (unsigned)(nt >> 3);
    const rsrc_t rO = RSRC(P.out[dst < NRANK ? dst : 0], dst < NRANK ? 0x2000000u : 0u);
    const unsigned ob = (unsigned)((rank * ML + mt * 256 + wid * 64) * 4096 + (nt & 7) * 512 + rot_of(RL, xcd) * 64);
    const int l2 = lane_now(), ov = (l2 & 15) * 4096 + (l2 >> 4) * 16;
#pragma unroll
    for (int m = 0; m < 4; ++m)
#pragma unroll
      for (int t = 0; t < OL; ++t) {
#pragma unroll
        for (int b = 1; b < SL; ++b) acc[m][t] += acc[m][b * OL + t];
        if (bad) acc[m][t] = (f4){__builtin_nanf(""), __builtin_nanf(""), __builtin_nanf(""), __builtin_nanf("")};
        ST128(ASU4(acc[m][t]), rO, ov + t * 64, ob + m * (16 * 4096), 17);
      }
    WAIT_VM0();
    if (wid == 0) STAMP(2);
    BAR();
  }
}

}

extern "C" void kernel_launch(int rank, int nranks, ncclComm_t comm,
                              hipStream_t stream,
                              void* const* d_in, const int* in_sizes, int n_in,
                              void* d_out, int out_size,
                              const int* in_dtypes, int out_dtype,
                              void* const* d_out_all,
                              void* const* d_ws_all, size_t ws_bytes) {
  if (nranks != NRANK || ws_bytes < WS_NEED || n_in < 2 || out_size != ND * ML) {
    fprintf(stderr, "unsupported configuration\n");
    return;
  }
  HIP_CHECK(hipFuncSetAttribute((const void*)fused_gemm_a2a, hipFuncAttributeMaxDynamicSharedMemorySize, LDS_BYTES));
  Params P;
  P.x = (const float*)d_in[0]; P.W = (const float*)d_in[1];
  for (int p = 0; p < NRANK; ++p) { P.out[p] = (char*)d_out_all[p]; P.ws[p] = (char*)d_ws_all[p]; }
  P.rank = rank;
  hipLaunchKernelGGL(fused_gemm_a2a, dim3(NWG), dim3(512), LDS_BYTES, stream, P);
  HIP_CHECK(hipGetLastError());
}
```

```cpp
#include <hip/hip_runtime.h>
#include <rccl/rccl.h>
#include <cstdio>

#define HIP_CHECK(expr) do { hipError_t _e = (expr); if (_e != hipSuccess) { \
    fprintf(stderr, "HIP error %s at %s:%d\n", hipGetErrorString(_e), __FILE__, __LINE__); \
    return; } } while (0)

namespace {

typedef float f4 __attribute__((ext_vector_type(4)));
typedef _Float16 h8 __attribute__((ext_vector_type(8)));
typedef _Float16 h2 __attribute__((ext_vector_type(2)));
typedef unsigned u4 __attribute__((ext_vector_type(4)));
typedef __amdgpu_buffer_rsrc_t rsrc_t;
#define LAS __attribute__((address_space(3)))
#define GAS __attribute__((address_space(1)))

constexpr int NRANK = 8, ML = 1024, KD = 8192, ND = 8192;
#define PLAN 2
constexpr int NRND = 3; constexpr unsigned LS_PACK = 0x221;
constexpr int NWG = 256, NSTEP = 128;
constexpr size_t WS_XT = 1u << 20, XT_BYTES = (size_t)ML * KD * 2;
constexpr size_t WS_PART = WS_XT + XT_BYTES, IMG_BYTES = 131072, ROUND_BYTES = NWG * IMG_BYTES, PART_BYTES = NRND * ROUND_BYTES;
constexpr size_t WS_NEED = WS_PART + PART_BYTES + (1u << 20);
constexpr int C_TICKET = 0, C_TMO = 32, C_RDONE = 64, C_FLAG = 8 * 32, C_CNT = 16 * 32, C_XRDY = C_CNT + 5 * 256 * 32;
constexpr unsigned long long DEADLINE = 100000000ull;
constexpr int SLOT = 16384, L_SEND = 2 * SLOT, L_CALL = L_SEND + 114688, L_PROG = L_CALL + 16, L_PF = L_CALL + 64, LDS_BYTES = L_PF + 256;
#define PF_DIST 5
#define S_RATE 30
#define XSP 1
#define P_PRIO 0
#ifndef M_PRIO
#define M_PRIO 1
#endif
#define H_PRIO 2
#define R_UB 16
#define SIG_AT 0
#define F_SLEEP 0
#define F_WIN 1
#define S_WIN2 2
#define E_FENCE 1
#define S_WIN 5
#define S_CATCH 2
#define K_REV 0
#define X_AUX 16

struct Params {
  const float* x; const float* W;
  char* out[NRANK]; char* ws[NRANK];
  int rank;
};

#define SB0() __builtin_amdgcn_sched_barrier(0)
#define SGB(m, n) __builtin_amdgcn_sched_group_barrier(m, n, 0)
#define ASF4(r) __builtin_bit_cast(f4, r)
#define ASU4(r) __builtin_bit_cast(u4, r)
#define ASH8(r) __builtin_bit_cast(h8, r)
#define BAR() do { __builtin_amdgcn_s_barrier(); SB0(); } while (0)
#define BARL() do { asm volatile("s_waitcnt lgkmcnt(0)" ::: "memory"); __builtin_amdgcn_s_barrier(); SB0(); } while (0)
#define WAIT_VM0() asm volatile("s_waitcnt vmcnt(0)" ::: "memory")
#define ST128(v, r, vo, so, aux) do { const u4 d_ = (v); __builtin_amdgcn_raw_buffer_store_b128(d_, r, vo, so, aux); asm volatile("s_nop 1" :: "v"(d_) : "memory"); } while (0)
#define RSRC(p, n) __builtin_amdgcn_make_buffer_rsrc((void*)(p), 0, (n), 0x00020000)

__device__ __forceinline__ unsigned pk(float a, float b) { h2 v = {(_Float16)a, (_Float16)b}; return __builtin_bit_cast(unsigned, v); }
__device__ __forceinline__ int lane_now() {
  int l; asm volatile("v_mbcnt_lo_u32_b32 %0, -1, 0\n\tv_mbcnt_hi_u32_b32 %0, -1, %0" : "=v"(l)); return l;
}
__device__ __forceinline__ int vm_now() {
  unsigned v; asm volatile("s_getreg_b32 %0, hwreg(HW_REG_IB_STS)" : "=s"(v) :: "memory");
  return (int)((v & 15u) | ((v >> 18) & 0x30u));
}
__host__ __device__ constexpr int ls_of(int r) { return (int)((LS_PACK >> (4 * r)) & 15u); }
__host__ __device__ constexpr int g_beg(int r) { return r == 0 ? 0 : g_beg(r - 1) + (128 >> ls_of(r - 1)); }
__device__ __forceinline__ int g_beg_rt(int r) {
  int v = 0;
#pragma unroll
  for (int i = 0; i < NRND; ++i) v = r > i ? g_beg(i + 1) : v;
  return v;
}
__device__ __forceinline__ int split_of(int r, int xcd) { return xcd >> (3 - ls_of(r)); }
#define MAP 0
__device__ __forceinline__ int tile_idx(int r, int xcd, int jq) { return (g_beg_rt(r) >> 1) + (xcd & ((8 >> ls_of(r)) - 1)) * 8 + jq; }
__device__ __forceinline__ int n_tile(int r, int rank, int xcd, int jq) {
  return ((rank + jq) & 7) * 8 + (g_beg_rt(r) >> 4) + (xcd & ((8 >> ls_of(r)) - 1));
}
#define K_ILV 0
__device__ __forceinline__ unsigned k_stride(int r, unsigned unit) {
  const unsigned st = K_ILV ? unit << ls_of(r) : unit;
  return r >= NRND ? 0u : (K_REV ? 0u - st : st);
}
__device__ __forceinline__ unsigned k_base(int r, int xcd, unsigned unit) {
  if (r >= NRND) return 0x20000000u;
  const int k0 = K_ILV ? split_of(r, xcd) - (g_beg_rt(r) << ls_of(r))
                       : ((split_of(r, xcd) + r) & ((1 << ls_of(r)) - 1)) * (128 >> ls_of(r)) - g_beg_rt(r);
  return (unsigned)(K_REV ? 127 - k0 : k0) * unit;
}
__device__ __forceinline__ unsigned w_base(int r, int rank, int xcd, int jq) {
  return k_base(r, xcd, 64u * ND * 4u) + (r >= NRND ? 0u : (unsigned)n_tile(r, rank, xcd, jq) * 512u);
}
__device__ __forceinline__ int rot_of(int r, int xcd) { return r >= NRND ? 0 : split_of(r, xcd) * (8 >> ls_of(r)); }
__device__ __forceinline__ unsigned ld_agent(const GAS unsigned* p) { return __hip_atomic_load(p, __ATOMIC_RELAXED, __HIP_MEMORY_SCOPE_AGENT); }

#define STAMP(i) do {} while (0)

__global__ void __launch_bounds__(512, 2) fused_gemm_a2a(const Params P) {
  extern __shared__ __attribute__((aligned(16))) char smem_[];
  LAS char* smem = (LAS char*)smem_;
  const int tid = threadIdx.x, lane = tid & 63, wid = __builtin_amdgcn_readfirstlane(tid >> 6);
  const int rank = P.rank;
  char* const ws = P.ws[rank];

  if (wid == 6) STAMP(0);
  const int xcd = blockIdx.x & 7, q = blockIdx.x >> 3, mt = q & 3, jq = q >> 2;
  GAS unsigned* const ctl = (GAS unsigned*)ws;
  constexpr unsigned WSTEP = 64u * ND * 4u;
  constexpr int NBAR = NSTEP;
  const rsrc_t rW = RSRC(P.W, 0x10000000u);
  const rsrc_t rP = RSRC(ws + WS_PART, (unsigned)PART_BYTES);

  const rsrc_t rX = RSRC(ws + WS_XT, (unsigned)XT_BYTES);
  {
    const int mb = blockIdx.x & 63, r8 = lane >> 3, c8 = lane & 7;
    f4 v[2][8];
#pragma unroll
    for (int h = 0; h < 2; ++h) {
      const float* src = P.x + (size_t)(mb * 16 + r8) * KD + ((blockIdx.x >> 6) * 16 + wid * 2 + h) * 128 + c8 * 4;
#pragma unroll
      for (int i = 0; i < 8; ++i) v[h][i] = *(const f4*)(src + (i & 1) * 8 * KD + (i >> 1) * 32);
    }
    typedef unsigned u2 __attribute__((ext_vector_type(2)));
    LAS char* tp = smem + L_SEND + wid * 8192;
#pragma unroll
    for (int h = 0; h < 2; ++h)
#pragma unroll
      for (int i = 0; i < 8; ++i)
        *(LAS u2*)(tp + (h * 4 + (i >> 1)) * 1024 + ((c8 >> 1) * 16 + (i & 1) * 8 + r8) * 16 + (c8 & 1) * 8) = (u2){pk(v[h][i][0], v[h][i][1]), pk(v[h][i][2], v[h][i][3])};
    if (wid == 6) STAMP(5);
#pragma unroll
    for (int h = 0; h < 2; ++h)
#pragma unroll
      for (int i = 0; i < 4; ++i)
        ST128(*(const LAS u4*)(tp + (h * 4 + i) * 1024 + lane * 16), rX, lane * 16,
              (unsigned)((((mb >> 2) * 256 + ((blockIdx.x >> 6) * 16 + wid * 2 + h) * 4 + i) * 4 + (mb & 3)) * 1024), 16);
    WAIT_VM0();
    if (wid == 6) STAMP(6);
    BARL();
  }
  if (wid == 6) STAMP(19);

  if (wid >= 6) __builtin_amdgcn_s_setprio(H_PRIO);
  if (wid == 7) {
#define COUNT_X(h) do { if (lane < 8) __hip_atomic_fetch_add(ctl + C_XRDY + ((((blockIdx.x >> 4) & 3) * 2 + (h)) * 8 + lane) * 32, 1u, __ATOMIC_RELAXED, __HIP_MEMORY_SCOPE_AGENT); } while (0)
    COUNT_X(0);
    unsigned ticket = 0;
    if (lane == 0) ticket = __hip_atomic_fetch_add(ctl + C_TICKET, 1u, __ATOMIC_RELAXED, __HIP_MEMORY_SCOPE_AGENT);
    const unsigned call = __builtin_amdgcn_readfirstlane(ticket) / NWG + 1;
    if (lane == 0) { *(volatile LAS unsigned*)(smem + L_CALL) = call; *(volatile LAS unsigned*)(smem + L_PROG) = 0; }
    const int pv = (mt * 16 + (lane >> 2)) * (ND * 4) + (lane & 3) * 128;
    unsigned wb[NRND];
#pragma unroll
    for (int r = 0; r < NRND; ++r) wb[r] = w_base(r, rank, xcd, jq);
    const unsigned pfl = (unsigned)(uintptr_t)(smem + L_PF);
#define PREFETCH(g) do { unsigned so_ = wb[0], st_ = k_stride(0, WSTEP), keep_; \
      _Pragma("unroll") for (int i_ = 1; i_ < NRND; ++i_) { so_ = (g) >= g_beg(i_) ? wb[i_] : so_; st_ = (g) >= g_beg(i_) ? k_stride(i_, WSTEP) : st_; } \
      so_ = __builtin_amdgcn_readfirstlane(so_ + (unsigned)(g) * st_); \
      if ((g) < NSTEP) asm volatile("s_mov_b32 %0, m0\n\ts_mov_b32 m0, %1\n\ts_nop 3\n\tbuffer_load_dword %2, %3, %4 offen lds\n\ts_mov_b32 m0, %0" \
        : "=&s"(keep_) : "s"(pfl), "v"(pv), "s"(rW), "s"(so_) : "memory"); } while (0)
#define TICK_S() do { if (nb < NBAR) { BAR(); ++nb; PREFETCH(nb + PF_DIST); } else __builtin_amdgcn_s_sleep(4); } while (0)
    for (int g = 3; g <= PF_DIST; ++g) PREFETCH(g);
#define WAIT_X(h) do { const unsigned long long t0_ = __builtin_amdgcn_s_memrealtime(); \
      while (__builtin_amdgcn_readfirstlane(ld_agent(ctl + C_XRDY + ((mt * 2 + (h)) * 8 + xcd) * 32)) < call * 64u) { \
        if (ld_agent(ctl + C_TMO) == call || __builtin_amdgcn_s_memrealtime() - t0_ > DEADLINE) { \
          __hip_atomic_store(ctl + C_TMO, call, __ATOMIC_RELAXED, __HIP_MEMORY_SCOPE_AGENT); break; } \
        __builtin_amdgcn_s_sleep(2); } } while (0)
    STAMP(10);
    WAIT_X(0);
    STAMP(11);
    BARL();
    int nb = 0;

    int credit = 0;
    for (int r = 0; r < NRND - 1; ++r) {
      const int nt = n_tile(r, rank, xcd, jq), lt = 3 - ls_of(r);
      const unsigned dst = (unsigned)(nt >> 3);
      const rsrc_t rO = RSRC(P.out[dst < NRANK ? dst : 0], dst < NRANK ? 0x2000000u : 0u);
      const unsigned ob = (unsigned)((rank * ML + mt * 256) * 4096 + (nt & 7) * 512 + rot_of(r, xcd) * 64);
      const int srow = lane >> (2 + lt), sc = lane & ((4 << lt) - 1);
      const int ub = g_beg_rt(r);
      const int lb = L_SEND + ub * 1024 + (sc >> 2) * 1024 + ((sc & 3) * 16 + srow) * 16, ov = srow * 4096 + sc * 16;
      const int lsub = 256 >> lt, osub = (16 >> lt) * 4096, lm = (1 << lt) - 1;
      const int nu = 16 << lt;
      bool first = true;
      const int rate = S_RATE;
      for (int u = 0; u < nu;) {
        const unsigned pg = *(volatile LAS unsigned*)(smem + L_PROG);
        const int u1 = u + 1 < nu ? u + 1 : u;
        const u4 v0 = *(const volatile LAS u4*)(smem + lb + ((u >> lt) << (10 + lt)) + (u & lm) * lsub);
        const u4 v1 = *(const volatile LAS u4*)(smem + lb + ((u1 >> lt) << (10 + lt)) + (u1 & lm) * lsub);
        SB0();
        TICK_S();
        credit = min(credit + rate, 40);
        const int have = (int)__builtin_amdgcn_readfirstlane(pg) - ub, room = (nb < NBAR ? S_WIN : S_WIN2) - vm_now();
        if (have > u && credit >= 16 && room > 0) {
          if (first) { STAMP(8 + r); first = false; }
          ST128(v0, rO, ov, ob + (unsigned)((u >> lt) * 16 * 4096 + (u & lm) * osub), 17);
          credit -= 16; ++u;
          if (u < nu && have > u && credit >= 16 && room > 1) {
            ST128(v1, rO, ov, ob + (unsigned)((u >> lt) * 16 * 4096 + (u & lm) * osub), 17);
            credit -= 16; ++u;
          }
        }
      }
      STAMP(12 + r);
    }
    while (nb < NBAR) TICK_S();
    WAIT_VM0();
    BAR();
    STAMP(16);
    unsigned old = 0;
    if (lane == 0) old = __hip_atomic_fetch_add(ctl + C_RDONE, 1u, E_FENCE == 2 ? __ATOMIC_RELEASE : __ATOMIC_RELAXED, __HIP_MEMORY_SCOPE_AGENT);
    old = __builtin_amdgcn_readfirstlane(old);
    if ((old & (NWG - 1)) == NWG - 1) {
      if (E_FENCE == 2) __builtin_amdgcn_fence(__ATOMIC_ACQUIRE, "agent");
      if (E_FENCE >= 1) __builtin_amdgcn_fence(__ATOMIC_RELEASE, "");
      WAIT_VM0();
      const bool peer = lane < NRANK && lane != rank;
      char* pw = P.ws[0];
#pragma unroll
      for (int p = 1; p < NRANK; ++p) pw = lane == p ? P.ws[p] : pw;
      if (peer) __hip_atomic_store((GAS unsigned*)pw + C_FLAG + rank * 32, call, __ATOMIC_RELAXED, __HIP_MEMORY_SCOPE_SYSTEM);
      STAMP(17);
      const unsigned long long t1 = __builtin_amdgcn_s_memrealtime();
      for (;;) {
        unsigned f = call;
        if (peer) f = __hip_atomic_load(ctl + C_FLAG + lane * 32, __ATOMIC_RELAXED, __HIP_MEMORY_SCOPE_SYSTEM);
        if (__all(f >= call)) break;
        if (ld_agent(ctl + C_TMO) == call || __builtin_amdgcn_s_memrealtime() - t1 > DEADLINE) {
          __hip_atomic_store(ctl + C_TMO, call, __ATOMIC_RELAXED, __HIP_MEMORY_SCOPE_AGENT);
          if (lane == 0) ((float*)P.out[rank])[(size_t)rank * ML * 1024] = __builtin_nanf("");
          break;
        }
        __builtin_amdgcn_s_sleep(2);
      }
      __builtin_amdgcn_fence(__ATOMIC_ACQUIRE, "");
      STAMP(18);
    }
    return;
  }

  if (wid == 6) {
    BAR();
    const unsigned call = __builtin_amdgcn_readfirstlane(*(volatile LAS unsigned*)(smem + L_CALL));
    int nb = 0; bool bad = false;
    unsigned long long t0 = 0;
#define TICK_R() do { if (nb < NBAR) { BAR(); ++nb; } else { \
      if (t0 == 0) t0 = __builtin_amdgcn_s_memrealtime(); \
      else if (__builtin_amdgcn_s_memrealtime() - t0 > DEADLINE) bad = true; \
      __builtin_amdgcn_s_sleep(1); } } while (0)
#define REDUCE(r) do { constexpr int S_ = 1 << ls_of(r), OW_ = 8 / S_, UB_ = (R_UB / (S_ - 1) / OW_ > 0 ? R_UB / (S_ - 1) / OW_ : 1) * OW_, NB_ = 128 / S_ / UB_, SL_ = 131072 / S_; \
      while (nb < g_beg((r) + 1) + SIG_AT + 2) TICK_R(); \
      for (;;) { \
        const unsigned c_ = ld_agent(ctl + C_CNT + ((r) * NWG + blockIdx.x) * 32 + (lane & 3) * 8), tm_ = ld_agent(ctl + C_TMO); \
        do { TICK_R(); } while (vm_now() != 0); \
        if (__builtin_amdgcn_readfirstlane(tm_) == call) bad = true; \
        if (__all(c_ >= call * (S_ - 1))) break; \
        if (bad) { __hip_atomic_store(ctl + C_TMO, call, __ATOMIC_RELAXED, __HIP_MEMORY_SCOPE_AGENT); break; } \
      } \
      STAMP(25 + (r)); \
      const unsigned pb_ = (unsigned)(r) * (unsigned)ROUND_BYTES + blockIdx.x * (unsigned)IMG_BYTES; \
      u4 ld_[UB_][S_ - 1]; f4 own_[UB_]; \
      _Pragma("unroll 1") for (int b = 0; b < NB_; ++b) { \
        LAS f4* p_ = (LAS f4*)(smem + L_SEND + (g_beg(r) + b * UB_) * 1024 + lane * 16); \
        _Pragma("unroll") for (int i = 0; i < UB_; ++i) { own_[i] = p_[i * 64]; _Pragma("unroll") for (int e = 0; e < S_ - 1; ++e) \
          ld_[i][e] = __builtin_amdgcn_raw_buffer_load_b128(rP, lane * 16, pb_ + e * SL_ + (b * UB_ + i) * 1024, 16); } \
        SB0(); \
        do { TICK_R(); } while (vm_now() != 0); \
        _Pragma("unroll") for (int i = 0; i < UB_; ++i) { \
          f4 a_ = own_[i]; \
          _Pragma("unroll") for (int e = 0; e < S_ - 1; ++e) a_ += ASF4(ld_[i][e]); \
          if (bad) a_ = (f4){__builtin_nanf(""), __builtin_nanf(""), __builtin_nanf(""), __builtin_nanf("")}; \
          p_[i * 64] = a_; } \
        asm volatile("" ::: "memory"); \
        *(volatile LAS unsigned*)(smem + L_PROG) = g_beg(r) + (b + 1) * UB_; SB0(); \
      } } while (0)
    REDUCE(0);
    if constexpr (NRND > 2) REDUCE(NRND > 2 ? 1 : 0);
    if constexpr (NRND > 3) REDUCE(NRND > 3 ? 2 : 0);
    if constexpr (NRND > 4) REDUCE(NRND > 4 ? 3 : 0);
    while (nb < NBAR) { BAR(); ++nb; }
    return;
  }

  if (wid >= 4) {
    __builtin_amdgcn_s_setprio(P_PRIO);
    const int c = lane & 31, kg0 = (wid - 4) * 4 + (lane >> 5);
    const int wv = (kg0 * 8 * ND + c * 4) * 4;
    const int wrest = (kg0 >> 2) * 1024 + (kg0 & 3) * 256 + (c & 3) * 16;
    u4 wr[2][2][8];
#define WOFF(rot) ((((c >> 2) - (rot)) & 7) * 2048 + wrest + ((((c >> 2) - (rot)) & 1) << 6))
#define WLOADU(set, u, so) _Pragma("unroll") for (int j = 0; j < 8; ++j) \
      wr[set][u][j] = __builtin_amdgcn_raw_buffer_load_b128(rW, wv, (so) + ((u) * 16 + j) * (ND * 4), 0)
#define WCVT(set, u, i) (u4){pk(ASF4(wr[set][u][0])[i], ASF4(wr[set][u][1])[i]), pk(ASF4(wr[set][u][2])[i], ASF4(wr[set][u][3])[i]), \
                             pk(ASF4(wr[set][u][4])[i], ASF4(wr[set][u][5])[i]), pk(ASF4(wr[set][u][6])[i], ASF4(wr[set][u][7])[i])}
#define WWRITEU(set, u, buf, wo) _Pragma("unroll") for (int i = 0; i < 4; ++i) \
      *(LAS u4*)(smem + (buf) * SLOT + ((wo) ^ ((i & 1) << 6)) + (i >> 1) * 128 + (u) * 512) = WCVT(set, u, i)
    unsigned cur = w_base(0, rank, xcd, jq);
    int wcur = WOFF(rot_of(0, xcd));
    unsigned scur = k_stride(0, WSTEP);
    WLOADU(0, 0, cur); WLOADU(0, 1, cur); SB0(); WLOADU(1, 0, cur + scur); WLOADU(1, 1, cur + scur); SB0();
    WWRITEU(0, 0, 0, wcur); WWRITEU(0, 1, 0, wcur); WLOADU(0, 0, cur + 2 * scur); WLOADU(0, 1, cur + 2 * scur); SB0();
    BARL();
    for (int r = 0; r < NRND; ++r) {
      const int ge = g_beg_rt(r + 1);
      const unsigned nxt = w_base(r + 1, rank, xcd, jq);
      const unsigned snxt = k_stride(r + 1, WSTEP);
      const int wnxt = WOFF(rot_of(r + 1, xcd));
      for (int g = g_beg_rt(r); g < ge; g += 2) {
#pragma unroll
        for (int z = 0; z < 2; ++z) {
          const unsigned wso = g + z + 3 < ge ? cur + (unsigned)(g + z + 3) * scur : nxt + (unsigned)(g + z + 3) * snxt;
          const int wo = g + z + 1 < ge ? wcur : wnxt;
          WWRITEU((z + 1) & 1, 0, (z + 1) & 1, wo); SB0(); WLOADU((z + 1) & 1, 0, wso); SB0();
          WWRITEU((z + 1) & 1, 1, (z + 1) & 1, wo); SB0(); WLOADU((z + 1) & 1, 1, wso); SB0();
          BARL();
        }
      }
      cur = nxt; scur = snxt; wcur = wnxt;
    }
    return;
  }

  __builtin_amdgcn_s_setprio(M_PRIO);
  const int rcp = (lane >> 2) & 3;
  const int rd0 = (int)(uintptr_t)smem + (lane >> 4) * 256 + (rcp | (((lane & 1) ^ 0) << 2) | ((lane & 2) << 2)) * 16;
  const int rd1 = (int)(uintptr_t)smem + (lane >> 4) * 256 + (rcp | (((lane & 1) ^ 1) << 2) | ((lane & 2) << 2)) * 16;
  const int xv = lane * 16;
  const unsigned xs0 = (unsigned)(mt * 4 + wid) * (256u * 4096u);
  f4 acc[4][8] = {};
  u4 xf[4][4], wf[8] = {};
#define XLOAD(j, so) _Pragma("unroll") for (int m = 0; m < 4; ++m) xf[j][m] = __builtin_amdgcn_raw_buffer_load_b128(rX, xv + m * 1024, (so), X_AUX)
#define FRD(t, buf, kk) asm volatile("ds_read_b128 %0, %1 offset:%2" : "+v"(wf[t]) : "v"(((t) & 1) ? rd1 : rd0), "n"((buf) * SLOT + ((t) * 2 + (kk)) * 1024) : "memory")
#define WAIT_L(n) asm volatile("s_waitcnt lgkmcnt(" #n ")" ::: "memory")
#define BW0()
#define BW1()
#define XW()
#define IW0()
#define IW1()
#define LW(t)
#define MM(t, j, z) _Pragma("unroll") for (int m = 0; m < 4; ++m) { \
    if (z) asm volatile("v_mfma_f32_16x16x32_f16 %0, %1, %2, 0" : "+v"(acc[m][t]) : "v"(wf[t]), "v"(xf[j][m])); \
    else asm volatile("v_mfma_f32_16x16x32_f16 %0, %1, %2, %0" : "+v"(acc[m][t]) : "v"(wf[t]), "v"(xf[j][m])); }
#define XLD(j, m, so) xf[j][m] = __builtin_amdgcn_raw_buffer_load_b128(rX, xv + (m) * 1024, (so), X_AUX)
#define STEP(buf, sa, sb, z) do { \
    _Pragma("unroll") for (int t = 0; t < 8; ++t) { LW(t); WAIT_L(7); MM(t, (buf) * 2, z); FRD(t, buf, 1); SB0(); \
      if (XSP ? (t & 1) : t == 7) { if (XSP) { IW0(); XLD(((buf) * 2 + 3) & 3, t >> 1, sa); IW1(); } else { XLOAD((buf) * 2, sb); } SB0(); } } \
    WAIT_L(7); MM(0, (buf) * 2 + 1, 0); WAIT_L(6); MM(1, (buf) * 2 + 1, 0); SB0(); if (XSP) XLD((buf) * 2, 0, sb); SB0(); \
    BW0(); BARL(); BW1(); \
    FRD(0, (buf) ^ 1, 0); FRD(1, (buf) ^ 1, 0); SB0(); \
    _Pragma("unroll") for (int t = 2; t < 8; ++t) { MM(t, (buf) * 2 + 1, 0); FRD(t, (buf) ^ 1, 0); SB0(); \
      if (XSP ? (t & 1) : t == 7) { if (XSP) XLD((buf) * 2, t >> 1, sb); else { XLOAD((buf) * 2 + 1, sb + 4096u); } SB0(); } } } while (0)
#define PARTNER(r, b) (q * 8 + ((((split_of(r, xcd) + (b)) & ((1 << ls_of(r)) - 1)) << (3 - ls_of(r))) | (xcd & ((8 >> ls_of(r)) - 1))))
#define PUT_PARTIALS(r) do { constexpr int S_ = 1 << ls_of(r), OW_ = 8 / S_; \
    _Pragma("unroll") for (int b = 1; b < S_; ++b) { \
      const unsigned o_ = (unsigned)(r) * (unsigned)ROUND_BYTES + wid * (4096 * OW_) + (unsigned)PARTNER(r, b) * (unsigned)IMG_BYTES + (b - 1) * (16384 * OW_); \
      _Pragma("unroll") for (int m = 0; m < 4; ++m) _Pragma("unroll") for (int t = 0; t < OW_; ++t) \
        ST128(ASU4(acc[m][b * OW_ + t]), rP, xv + t * 1024, o_ + m * (OW_ * 1024), 16); } } while (0)
#define SIGNAL(r) do { const int l_ = lane_now(); if (l_ >= 1 && l_ < (1 << ls_of(r))) \
    __hip_atomic_fetch_add(ctl + C_CNT + ((r) * NWG + PARTNER(r, l_)) * 32 + wid * 8, 1u, __ATOMIC_RELAXED, __HIP_MEMORY_SCOPE_AGENT); } while (0)
#define XOFF(r, g) (((g) < ge ? cur : nxt) + (unsigned)(g) * ((g) < ge ? k_stride(r, 8192u) : k_stride((r) + 1, 8192u)))
#define PAIR(r, g, z) do { const unsigned s1 = XOFF(r, (g) + 1), s2 = XOFF(r, (g) + 2), s3 = XOFF(r, (g) + 3); \
    STEP(0, s1 + 4096u, s2, z); STEP(1, s2 + 4096u, s3, 0); } while (0)
#define ROUND(r) do { \
    const int ge = g_beg((r) + 1); const unsigned nxt = xs0 + k_base((r) + 1, xcd, 8192u); \
    PAIR(r, g_beg(r), 1); \
    if ((r) > 0) {     \
      asm volatile("s_waitcnt vmcnt(16)" ::: "memory"); SIGNAL((r) > 0 ? (r) - 1 : 0); SB0(); if (wid == 0) STAMP(2 + (r)); } \
    _Pragma("unroll 1") for (int g = g_beg(r) + 2; g < ge; g += 2) PAIR(r, g, 0); \
    asm volatile("s_nop 15" ::: "memory"); SB0();     \
    cur = nxt; if (wid == 0) STAMP(20 + (r)); } while (0)
  unsigned cur = xs0 + k_base(0, xcd, 8192u);
  BAR();
  XLOAD(0, cur); XLOAD(1, cur + 4096); XLOAD(2, cur + k_stride(0, 8192u)); if (!XSP) { XLOAD(3, cur + k_stride(0, 8192u) + 4096); } SB0();
  unsigned call = __builtin_amdgcn_readfirstlane(*(volatile LAS unsigned*)(smem + L_CALL));
  asm volatile("" : "+s"(call)); SB0();
  bool bad = false;
#pragma unroll
  for (int t = 0; t < 8; ++t) FRD(t, 0, 0);
  SB0();
  STAMP(1);
#define FULL_ROUND(r) do { constexpr int OW_ = 8 >> ls_of(r); ROUND(r); PUT_PARTIALS(r); \
    _Pragma("unroll") for (int m = 0; m < 4; ++m) _Pragma("unroll") for (int t = 0; t < OW_; ++t) \
      *(LAS f4*)(smem + L_SEND + g_beg(r) * 1024 + wid * (4096 * OW_) + xv + (m * OW_ + t) * 1024) = acc[m][t]; \
    SB0(); } while (0)
  FULL_ROUND(0);
  if constexpr (NRND > 2) FULL_ROUND(NRND > 2 ? 1 : 0);
  if constexpr (NRND > 3) FULL_ROUND(NRND > 3 ? 2 : 0);
  if constexpr (NRND > 4) FULL_ROUND(NRND > 4 ? 3 : 0);
  constexpr int RL = NRND - 1, SL = 1 << ls_of(RL), OL = 8 / SL;
  ROUND(RL);
  PUT_PARTIALS(RL);
  WAIT_VM0();
  SIGNAL(RL);
  if (wid == 0) STAMP(7);
  { const GAS unsigned* cnt = ctl + C_CNT + (RL * NWG + blockIdx.x) * 32 + wid * 8;
    const unsigned long long t0 = __builtin_amdgcn_s_memrealtime();
    while (__builtin_amdgcn_readfirstlane(ld_agent(cnt)) < call * (SL - 1)) {
      if (ld_agent(ctl + C_TMO) == call || __builtin_amdgcn_s_memrealtime() - t0 > DEADLINE) {
        __hip_atomic_store(ctl + C_TMO, call, __ATOMIC_RELAXED, __HIP_MEMORY_SCOPE_AGENT); bad = true; break; }
      __builtin_amdgcn_s_sleep(1); } }
  if (ld_agent(ctl + C_TMO) == call) bad = true;
  if (wid == 0) STAMP(29);
#pragma unroll
  for (int b = 1; b < SL; ++b)
#pragma unroll
    for (int m = 0; m < 4; ++m)
#pragma unroll
      for (int t = 0; t < OL; ++t)
        acc[m][b * OL + t] = ASF4(__builtin_amdgcn_raw_buffer_load_b128(rP, xv + t * 1024,
            (unsigned)RL * (unsigned)ROUND_BYTES + blockIdx.x * (unsigned)IMG_BYTES + wid * (4096 * OL) + (b - 1) * (16384 * OL) + m * (OL * 1024), 16));
  {
    const int nt = n_tile(RL, rank, xcd, jq);
    const unsigned dst = (unsigned)(nt >> 3);
    const rsrc_t rO = RSRC(P.out[dst < NRANK ? dst : 0], dst < NRANK ? 0x2000000u : 0u);
    const unsigned ob = (unsigned)((rank * ML + mt * 256 + wid * 64) * 4096 + (nt & 7) * 512 + rot_of(RL, xcd) * 64);
    const int l2 = lane_now(), ov = (l2 & 15) * 4096 + (l2 >> 4) * 16;
#pragma unroll
    for (int m = 0; m < 4; ++m)
#pragma unroll
      for (int t = 0; t < OL; ++t) {
#pragma unroll
        for (int b = 1; b < SL; ++b) acc[m][t] += acc[m][b * OL + t];
        if (bad) acc[m][t] = (f4){__builtin_nanf(""), __builtin_nanf(""), __builtin_nanf(""), __builtin_nanf("")};
        ST128(ASU4(acc[m][t]), rO, ov + t * 64, ob + m * (16 * 4096), 17);
        if (F_WIN == 1) { WAIT_VM0(); if (F_SLEEP && jq != 0 && !(m == 3 && t == OL - 1)) __builtin_amdgcn_s_sleep(F_SLEEP); }
        if (F_WIN == 2) asm volatile("s_waitcnt vmcnt(1)" ::: "memory");
      }
    WAIT_VM0();
    if (wid == 0) STAMP(2);
    BAR();
  }
}

}

extern "C" void kernel_launch(int rank, int nranks, ncclComm_t comm,
                              hipStream_t stream,
                              void* const* d_in, const int* in_sizes, int n_in,
                              void* d_out, int out_size,
                              const int* in_dtypes, int out_dtype,
                              void* const* d_out_all,
                              void* const* d_ws_all, size_t ws_bytes) {
  if (nranks != NRANK || ws_bytes < WS_NEED || n_in < 2 || out_size != ND * ML) {
    fprintf(stderr, "unsupported configuration\n");
    return;
  }
  HIP_CHECK(hipFuncSetAttribute((const void*)fused_gemm_a2a, hipFuncAttributeMaxDynamicSharedMemorySize, LDS_BYTES));
  Params P;
  P.x = (const float*)d_in[0]; P.W = (const float*)d_in[1];
  for (int p = 0; p < NRANK; ++p) { P.out[p] = (char*)d_out_all[p]; P.ws[p] = (char*)d_ws_all[p]; }
  P.rank = rank;
  hipLaunchKernelGGL(fused_gemm_a2a, dim3(NWG), dim3(512), LDS_BYTES, stream, P);
  HIP_CHECK(hipGetLastError());
}
```

```cpp
#include <hip/hip_runtime.h>
#include <rccl/rccl.h>
#include <cstdio>

#define HIP_CHECK(expr) do { hipError_t _e = (expr); if (_e != hipSuccess) { \
    fprintf(stderr, "HIP error %s at %s:%d\n", hipGetErrorString(_e), __FILE__, __LINE__); \
    return; } } while (0)

namespace {

typedef float f4 __attribute__((ext_vector_type(4)));
typedef _Float16 h2 __attribute__((ext_vector_type(2)));
typedef unsigned u4 __attribute__((ext_vector_type(4)));
typedef unsigned u2 __attribute__((ext_vector_type(2)));
typedef __amdgpu_buffer_rsrc_t rsrc_t;
#define LAS __attribute__((address_space(3)))
#define GAS __attribute__((address_space(1)))

constexpr int NRANK = 8, ML = 1024, KD = 8192, ND = 8192;
constexpr int NRND = 3; constexpr unsigned LS_PACK = 0x221;
constexpr int NWG = 256, NSTEP = 128;
constexpr size_t WS_XT = 1u << 20, XT_BYTES = (size_t)ML * KD * 2;
constexpr size_t WS_PART = WS_XT + XT_BYTES, IMG_BYTES = 131072, ROUND_BYTES = NWG * IMG_BYTES, PART_BYTES = NRND * ROUND_BYTES;
constexpr size_t WS_NEED = WS_PART + PART_BYTES + (1u << 20);
constexpr int C_TICKET = 0, C_TMO = 32, C_RDONE = 64, C_FLAG = 8 * 32, C_CNT = 16 * 32, C_XRDY = C_CNT + NRND * NWG * 32;
constexpr unsigned long long DEADLINE = 100000000ull;
constexpr int SLOT = 16384, L_SEND = 2 * SLOT, L_CALL = L_SEND + 98304, L_PROG = L_CALL + 16, L_PF = L_CALL + 64, LDS_BYTES = L_PF + 256;
#define PF_DIST 7
#define PF_FROM 76
#define S_GAP 68
#define S_STEP 70
#define S_WIN 5
#define S_WIN2 2

struct Params {
  const float* x; const float* W;
  char* out[NRANK]; char* ws[NRANK];
  int rank;
};

#define SB0() __builtin_amdgcn_sched_barrier(0)
#define ASF4(r) __builtin_bit_cast(f4, r)
#define ASU4(r) __builtin_bit_cast(u4, r)
#define BAR() do { __builtin_amdgcn_s_barrier(); SB0(); } while (0)
#define BARL() do { asm volatile("s_waitcnt lgkmcnt(0)" ::: "memory"); __builtin_amdgcn_s_barrier(); SB0(); } while (0)
#define WAIT_V(n) asm volatile("s_waitcnt vmcnt(" #n ")" ::: "memory")
#define WAIT_L(n) asm volatile("s_waitcnt lgkmcnt(" #n ")" ::: "memory")
#define ST128(v, r, vo, so, aux) do { const u4 d_ = (v); __builtin_amdgcn_raw_buffer_store_b128(d_, r, vo, so, aux); asm volatile("s_nop 1" :: "v"(d_) : "memory"); } while (0)
#define RSRC(p, n) __builtin_amdgcn_make_buffer_rsrc((void*)(p), 0, (n), 0x00020000)
#define NAN4 ((f4){__builtin_nanf(""), __builtin_nanf(""), __builtin_nanf(""), __builtin_nanf("")})

__device__ __forceinline__ unsigned pk(float a, float b) { h2 v = {(_Float16)a, (_Float16)b}; return __builtin_bit_cast(unsigned, v); }
__device__ __forceinline__ int lane_now() {
  int l; asm volatile("v_mbcnt_lo_u32_b32 %0, -1, 0\n\tv_mbcnt_hi_u32_b32 %0, -1, %0" : "=v"(l)); return l;
}
__device__ __forceinline__ int vm_now() {
  unsigned v; asm volatile("s_getreg_b32 %0, hwreg(HW_REG_IB_STS)" : "=s"(v) :: "memory");
  return (int)((v & 15u) | ((v >> 18) & 0x30u));
}
__host__ __device__ constexpr int ls_of(int r) { return (int)((LS_PACK >> (4 * r)) & 15u); }
constexpr unsigned long long g_pack() { unsigned long long v = 0; int g = 0; for (int r = 0; r <= NRND; ++r) { v |= (unsigned long long)g << (8 * r); g += 128 >> ls_of(r); } return v; }
constexpr unsigned long long G_PACK = g_pack();
__host__ __device__ constexpr int g_beg(int r) { return (int)((G_PACK >> (8 * r)) & 255u); }
__device__ __forceinline__ int split_of(int r, int xcd) { return xcd >> (3 - ls_of(r)); }
__device__ __forceinline__ int n_tile(int r, int rank, int xcd, int jq) {
  return ((rank + jq) & 7) * 8 + (g_beg(r) >> 4) + (xcd & ((8 >> ls_of(r)) - 1));
}
__device__ __forceinline__ int k_off(int r, int xcd) { return ((split_of(r, xcd) + r) & ((1 << ls_of(r)) - 1)) * (128 >> ls_of(r)) - g_beg(r); }
__device__ __forceinline__ unsigned w_base(int r, int rank, int xcd, int jq) {
  return r >= NRND ? 0x20000000u : (unsigned)k_off(r, xcd) * (64u * ND * 4u) + (unsigned)n_tile(r, rank, xcd, jq) * 512u;
}
__device__ __forceinline__ unsigned x_base(int r, int xcd) { return r >= NRND ? 0x20000000u : (unsigned)k_off(r, xcd) * 8192u; }
__device__ __forceinline__ int rot_of(int r, int xcd) { return split_of(r, xcd) * (8 >> ls_of(r)); }
__device__ __forceinline__ unsigned ld_agent(const GAS unsigned* p) { return __hip_atomic_load(p, __ATOMIC_RELAXED, __HIP_MEMORY_SCOPE_AGENT); }

#define STAMP(i) do {} while (0)

__global__ void __launch_bounds__(512, 2) fused_gemm_a2a(const Params P) {
  extern __shared__ __attribute__((aligned(16))) char smem_[];
  LAS char* smem = (LAS char*)smem_;
  const int tid = threadIdx.x, lane = tid & 63, wid = __builtin_amdgcn_readfirstlane(tid >> 6);
  const int rank = P.rank;
  char* const ws = P.ws[rank];
  if (wid == 6) STAMP(0);
  const int xcd = blockIdx.x & 7, q = blockIdx.x >> 3, mt = q & 3, jq = q >> 2;
  GAS unsigned* const ctl = (GAS unsigned*)ws;
  constexpr unsigned WSTEP = 64u * ND * 4u;
  const rsrc_t rW = RSRC(P.W, 0x10000000u);
  const rsrc_t rP = RSRC(ws + WS_PART, (unsigned)PART_BYTES);
  const rsrc_t rX = RSRC(ws + WS_XT, (unsigned)XT_BYTES);

  {
    const int mb = blockIdx.x & 63, r8 = lane >> 3, c8 = lane & 7, kq = (blockIdx.x >> 6) * 16 + wid * 2;
    f4 v[16];
    const float* src = P.x + (size_t)(mb * 16 + r8) * KD + kq * 128 + c8 * 4;
#pragma unroll
    for (int i = 0; i < 16; ++i) v[i] = __builtin_nontemporal_load((const f4*)(src + (i & 1) * 8 * KD + (i >> 1) * 32));
    LAS char* tp = smem + L_SEND + wid * 8192;
#pragma unroll
    for (int i = 0; i < 16; ++i)
      *(LAS u2*)(tp + (i >> 1) * 1024 + ((c8 >> 1) * 16 + (i & 1) * 8 + r8) * 16 + (c8 & 1) * 8) = (u2){pk(v[i][0], v[i][1]), pk(v[i][2], v[i][3])};
#pragma unroll
    for (int i = 0; i < 8; ++i)
      ST128(*(const LAS u4*)(tp + i * 1024 + lane * 16), rX, lane * 16, (unsigned)((((mb >> 2) * 256 + kq * 4 + i) * 4 + (mb & 3)) * 1024), 16);
    WAIT_V(0);
    BARL();
  }
  if (wid >= 6) __builtin_amdgcn_s_setprio(2);

  if (wid == 7) {
    if (lane < 8) __hip_atomic_fetch_add(ctl + C_XRDY + ((blockIdx.x >> 4) * 8 + lane) * 32, 1u, __ATOMIC_RELAXED, __HIP_MEMORY_SCOPE_AGENT);
    unsigned ticket = 0;
    if (lane == 0) ticket = __hip_atomic_fetch_add(ctl + C_TICKET, 1u, __ATOMIC_RELAXED, __HIP_MEMORY_SCOPE_AGENT);
    const unsigned call = __builtin_amdgcn_readfirstlane(ticket) / NWG + 1;
    if (lane == 0) { *(volatile LAS unsigned*)(smem + L_CALL) = call; *(volatile LAS unsigned*)(smem + L_PROG) = 0; }
    const int pv = (mt * 16 + (lane >> 2)) * (ND * 4) + (lane & 3) * 128;
    const unsigned wb1 = w_base(1, rank, xcd, jq), wb2 = w_base(2, rank, xcd, jq);
    static_assert(PF_FROM >= g_beg(1) && NRND == 3, "prefetch offsets");
    const unsigned pfl = (unsigned)(uintptr_t)(smem + L_PF);
#define PREFETCH(g) do { if ((g) >= PF_FROM && (g) < NSTEP) { unsigned keep_; \
      const unsigned so_ = __builtin_amdgcn_readfirstlane(((g) >= g_beg(2) ? wb2 : wb1) + (unsigned)(g) * WSTEP); \
      asm volatile("s_mov_b32 %0, m0\n\ts_mov_b32 m0, %1\n\ts_nop 3\n\tbuffer_load_dword %2, %3, %4 offen lds\n\ts_mov_b32 m0, %0" \
        : "=&s"(keep_) : "s"(pfl), "v"(pv), "s"(rW), "s"(so_) : "memory"); } } while (0)
#define TICK_S() do { if (nb < NSTEP) { BAR(); ++nb; PREFETCH(nb + PF_DIST); } else __builtin_amdgcn_s_sleep(4); } while (0)
#define WAIT_X(kq) do { const unsigned long long t0_ = __builtin_amdgcn_s_memrealtime(); \
      while (!__all(ld_agent(ctl + C_XRDY + (((kq) * 4 + mt) * 8 + xcd) * 32) >= call * 16u)) { \
        if (ld_agent(ctl + C_TMO) == call || __builtin_amdgcn_s_memrealtime() - t0_ > DEADLINE) { \
          __hip_atomic_store(ctl + C_TMO, call, __ATOMIC_RELAXED, __HIP_MEMORY_SCOPE_AGENT); break; } \
        __builtin_amdgcn_s_sleep(2); } } while (0)
    WAIT_X(k_off(0, xcd) >> 5);
    BARL();
    int nb = 0;
    while (nb < 4) TICK_S();
    WAIT_X(lane & 3);

    const unsigned dst = (unsigned)((rank + jq) & 7);
    const rsrc_t rO = RSRC(P.out[dst < NRANK ? dst : 0], dst < NRANK ? 0x2000000u : 0u);
    unsigned tnext = (unsigned)__builtin_amdgcn_s_memrealtime();
    for (int r = 0; r < NRND - 1; ++r) {
      const int lt = 3 - ls_of(r);
      const unsigned ob = (unsigned)((rank * ML + mt * 256) * 4096 + (n_tile(r, rank, xcd, jq) & 7) * 512 + rot_of(r, xcd) * 64);
      const int srow = lane >> (2 + lt), sc = lane & ((4 << lt) - 1);
      const int ub = g_beg(r);
      const int lb = L_SEND + ub * 1024 + (sc >> 2) * 1024 + ((sc & 3) * 16 + srow) * 16, ov = srow * 4096 + sc * 16;
      const int lsub = 256 >> lt, osub = (16 >> lt) * 4096, lm = (1 << lt) - 1;
      const int nu = 16 << lt;
      for (int u = 0; u < nu;) {
        const unsigned pg = *(volatile LAS unsigned*)(smem + L_PROG);
        const int u1 = u + 1 < nu ? u + 1 : u;
        const u4 v0 = *(const volatile LAS u4*)(smem + lb + ((u >> lt) << (10 + lt)) + (u & lm) * lsub);
        const u4 v1 = *(const volatile LAS u4*)(smem + lb + ((u1 >> lt) << (10 + lt)) + (u1 & lm) * lsub);
        const unsigned now = (unsigned)__builtin_amdgcn_s_memrealtime() + S_STEP;
        SB0();
        TICK_S();
        const int lag = (int)(now - tnext);
        if (lag > 2 * S_GAP) tnext = now - 2 * S_GAP;
        const int due = nb >= NSTEP || jq == 0 ? 2 : (lag >= S_GAP ? 2 : (lag >= 0 ? 1 : 0));
        const int n = min(min((int)__builtin_amdgcn_readfirstlane(pg) - ub - u, nu - u), min(due, (nb < NSTEP ? S_WIN : S_WIN2) - vm_now()));
        if (n > 0) {
          if (u == 0) STAMP(8 + r);
          ST128(v0, rO, ov, ob + (unsigned)((u >> lt) * 16 * 4096 + (u & lm) * osub), 17);
          if (n > 1) ST128(v1, rO, ov, ob + (unsigned)((u1 >> lt) * 16 * 4096 + (u1 & lm) * osub), 17);
          u += n; tnext += (unsigned)n * S_GAP;
        }
      }
      STAMP(12 + r);
    }
    while (nb < NSTEP) TICK_S();
    WAIT_V(0);
    BAR();
    unsigned old = 0;
    if (lane == 0) old = __hip_atomic_fetch_add(ctl + C_RDONE, 1u, __ATOMIC_RELAXED, __HIP_MEMORY_SCOPE_AGENT);
    old = __builtin_amdgcn_readfirstlane(old);
    if ((old & (NWG - 1)) == NWG - 1) {
      __builtin_amdgcn_fence(__ATOMIC_RELEASE, "");
      WAIT_V(0);
      const bool peer = lane < NRANK && lane != rank;
      char* pw = P.ws[0];
#pragma unroll
      for (int p = 1; p < NRANK; ++p) pw = lane == p ? P.ws[p] : pw;
      if (peer) __hip_atomic_store((GAS unsigned*)pw + C_FLAG + rank * 32, call, __ATOMIC_RELAXED, __HIP_MEMORY_SCOPE_SYSTEM);
      STAMP(17);
      const unsigned long long t1 = __builtin_amdgcn_s_memrealtime();
      for (;;) {
        unsigned f = call;
        if (peer) f = __hip_atomic_load(ctl + C_FLAG + lane * 32, __ATOMIC_RELAXED, __HIP_MEMORY_SCOPE_SYSTEM);
        if (__all(f >= call)) break;
        if (ld_agent(ctl + C_TMO) == call || __builtin_amdgcn_s_memrealtime() - t1 > DEADLINE) {
          __hip_atomic_store(ctl + C_TMO, call, __ATOMIC_RELAXED, __HIP_MEMORY_SCOPE_AGENT);
          if (lane == 0) ((float*)P.out[rank])[(size_t)rank * ML * 1024] = __builtin_nanf("");
          break;
        }
        __builtin_amdgcn_s_sleep(2);
      }
      __builtin_amdgcn_fence(__ATOMIC_ACQUIRE, "");
      STAMP(18);
    }
    return;
  }

  if (wid == 6) {
    BAR();
    const unsigned call = __builtin_amdgcn_readfirstlane(*(volatile LAS unsigned*)(smem + L_CALL));
    int nb = 0; bool bad = false;
    unsigned long long t0 = 0;
#define TICK_R() do { if (nb < NSTEP) { BAR(); ++nb; } else { \
      if (t0 == 0) t0 = __builtin_amdgcn_s_memrealtime(); \
      else if (__builtin_amdgcn_s_memrealtime() - t0 > DEADLINE) bad = true; \
      __builtin_amdgcn_s_sleep(1); } } while (0)
#define R_LDS(r, b) ((LAS f4*)(smem + L_SEND + (g_beg(r) + (b) * OW_) * 1024 + lane * 16))
#define R_ISSUE(r, sl, b) do { _Pragma("unroll") for (int i = 0; i < OW_; ++i) _Pragma("unroll") for (int e = 0; e < S_ - 1; ++e) \
        ld_[sl][i][e] = __builtin_amdgcn_raw_buffer_load_b128(rP, lane * 16, pb_ + e * (131072 / S_) + ((b) * OW_ + i) * 1024, 16); SB0(); } while (0)
#define R_OWN(r, b) do { _Pragma("unroll") for (int i = 0; i < OW_; ++i) own_[i] = R_LDS(r, b)[i * 64]; SB0(); } while (0)
#define R_SUM(r, sl, b, young) do { do { TICK_R(); } while (vm_now() > (young)); \
        _Pragma("unroll") for (int i = 0; i < OW_; ++i) { f4 a_ = own_[i]; \
          _Pragma("unroll") for (int e = 0; e < S_ - 1; ++e) a_ += ASF4(ld_[sl][i][e]); \
          if (bad) a_ = NAN4; \
          R_LDS(r, b)[i * 64] = a_; } \
        asm volatile("" ::: "memory"); \
        *(volatile LAS unsigned*)(smem + L_PROG) = g_beg(r) + ((b) + 1) * OW_; SB0(); } while (0)
#define REDUCE(r) do { constexpr int S_ = 1 << ls_of(r), OW_ = 8 / S_; \
      while (nb < g_beg((r) + 1) + 2) TICK_R(); \
      for (;;) { \
        const unsigned c_ = ld_agent(ctl + C_CNT + ((r) * NWG + blockIdx.x) * 32 + (lane & 3) * 8), tm_ = ld_agent(ctl + C_TMO); \
        do { TICK_R(); } while (vm_now() != 0); \
        if (__builtin_amdgcn_readfirstlane(tm_) == call) bad = true; \
        if (__all(c_ >= call * (S_ - 1))) break; \
        if (bad) { __hip_atomic_store(ctl + C_TMO, call, __ATOMIC_RELAXED, __HIP_MEMORY_SCOPE_AGENT); break; } \
      } \
      const unsigned pb_ = (unsigned)(r) * (unsigned)ROUND_BYTES + blockIdx.x * (unsigned)IMG_BYTES; \
      u4 ld_[2][OW_][S_ - 1]; f4 own_[OW_]; \
      R_ISSUE(r, 0, 0); TICK_R(); R_ISSUE(r, 1, 1); R_OWN(r, 0); \
      _Pragma("unroll 1") for (int b = 0; b < 14; b += 2) { \
        R_SUM(r, 0, b, OW_ * (S_ - 1)); R_ISSUE(r, 0, b + 2); R_OWN(r, b + 1); \
        R_SUM(r, 1, b + 1, OW_ * (S_ - 1)); R_ISSUE(r, 1, b + 3); R_OWN(r, b + 2); } \
      R_SUM(r, 0, 14, OW_ * (S_ - 1)); R_OWN(r, 15); \
      R_SUM(r, 1, 15, 0); } while (0)
    REDUCE(0);
    REDUCE(1);
    while (nb < NSTEP) TICK_R();
    return;
  }

  if (wid >= 4) {
    const int c = lane & 31, ua = ((wid - 4) * 2 + mt) & 3, ub = (ua + 1) & 3;
    const int va = ((ua * 16 + (lane >> 5) * 8) * ND + c * 4) * 4, vb = ((ub * 16 + (lane >> 5) * 8) * ND + c * 4) * 4;
    const int la = (ua >> 1) * 1024 + ((ua & 1) * 2 + (lane >> 5)) * 256, lb = (ub >> 1) * 1024 + ((ub & 1) * 2 + (lane >> 5)) * 256;
#define WOFF(rot) ((((c >> 2) - (rot)) & 7) * 2048 + (c & 3) * 16 + ((((c >> 2) - (rot)) & 1) << 6))
#define WLD(dst, v, so) do { const unsigned so__ = (so); _Pragma("unroll") for (int j = 0; j < 8; ++j) dst[j] = __builtin_amdgcn_raw_buffer_load_b128(rW, v, so__ + j * (ND * 4), 0); } while (0)
#define WWR(src, buf, wo) _Pragma("unroll") for (int i = 0; i < 4; ++i) \
      *(LAS u4*)(smem + (buf) * SLOT + ((wo) ^ ((i & 1) << 6)) + (i >> 1) * 128) = \
        (u4){pk(ASF4(src[0])[i], ASF4(src[1])[i]), pk(ASF4(src[2])[i], ASF4(src[3])[i]), pk(ASF4(src[4])[i], ASF4(src[5])[i]), pk(ASF4(src[6])[i], ASF4(src[7])[i])}
    unsigned wbs[NRND + 1]; int wos[NRND];
#pragma unroll
    for (int r = 0; r <= NRND; ++r) wbs[r] = w_base(r, rank, xcd, jq);
#pragma unroll
    for (int r = 0; r < NRND; ++r) wos[r] = WOFF(rot_of(r, xcd));
#define W_AT(g) ({ unsigned v_ = wbs[0]; _Pragma("unroll") for (int i_ = 1; i_ <= NRND; ++i_) v_ = (g) >= g_beg(i_) ? wbs[i_] : v_; v_ + (unsigned)(g) * WSTEP; })
#define WO_AT(g) ({ int v_ = wos[0]; _Pragma("unroll") for (int i_ = 1; i_ < NRND; ++i_) v_ = (g) >= g_beg(i_) ? wos[i_] : v_; v_; })
#define PRODUCER(DA) do { u4 wa[DA][8], wb_[2][8]; \
    _Pragma("unroll") for (int z = 0; z < DA; ++z) { WLD(wa[z], va, W_AT(z)); SB0(); if (z < 2) { WLD(wb_[z], vb, W_AT(z)); SB0(); } } \
    WWR(wa[0], 0, wos[0] + la); WWR(wb_[0], 0, wos[0] + lb); SB0(); WLD(wb_[0], vb, W_AT(2)); SB0(); WLD(wa[0], va, W_AT(DA)); SB0(); \
    BARL(); \
    for (int g = 0; g < NSTEP; g += DA) { _Pragma("unroll") for (int z = 0; z < DA; ++z) { \
      const int wo = WO_AT(g + z + 1); \
      WWR(wb_[(z + 1) & 1], (z + 1) & 1, wo + lb); SB0(); WLD(wb_[(z + 1) & 1], vb, W_AT(g + z + 3)); SB0(); \
      WWR(wa[(z + 1) % DA], (z + 1) & 1, wo + la); SB0(); WLD(wa[(z + 1) % DA], va, W_AT(g + z + 1 + DA)); SB0(); \
      BARL(); } } } while (0)
    if (wid == 4) PRODUCER(4); else PRODUCER(2);
    return;
  }

  __builtin_amdgcn_s_setprio(1);
  const int rcp = (lane >> 2) & 3;
  const int rd0 = (int)(uintptr_t)smem + (lane >> 4) * 256 + (rcp | (((lane & 1) ^ 0) << 2) | ((lane & 2) << 2)) * 16;
  const int rd1 = (int)(uintptr_t)smem + (lane >> 4) * 256 + (rcp | (((lane & 1) ^ 1) << 2) | ((lane & 2) << 2)) * 16;
  const int xv = lane * 16;
  const unsigned xs0 = (unsigned)(mt * 4 + wid) * (256u * 4096u);
  f4 acc[4][8] = {};
  u4 xf[4][4], wf[8] = {};
#define FRD(t, buf, kk) asm volatile("ds_read_b128 %0, %1 offset:%2" : "+v"(wf[t]) : "v"(((t) & 1) ? rd1 : rd0), "n"((buf) * SLOT + ((t) * 2 + (kk)) * 1024) : "memory")
#define MM(t, j, z) _Pragma("unroll") for (int m = 0; m < 4; ++m) { \
    if (z) asm volatile("v_mfma_f32_16x16x32_f16 %0, %1, %2, 0" : "+v"(acc[m][t]) : "v"(wf[t]), "v"(xf[j][m])); \
    else asm volatile("v_mfma_f32_16x16x32_f16 %0, %1, %2, %0" : "+v"(acc[m][t]) : "v"(wf[t]), "v"(xf[j][m])); }
#define XLD(j, m, so) xf[j][m] = __builtin_amdgcn_raw_buffer_load_b128(rX, xv + (m) * 1024, (so), 16)
#define STEP(buf, sa, sb, z) do { \
    _Pragma("unroll") for (int t = 0; t < 8; ++t) { WAIT_L(7); MM(t, (buf) * 2, z); FRD(t, buf, 1); SB0(); \
      if (t & 1) { XLD(((buf) * 2 + 3) & 3, t >> 1, sa); SB0(); } } \
    WAIT_L(7); MM(0, (buf) * 2 + 1, 0); WAIT_L(6); MM(1, (buf) * 2 + 1, 0); SB0(); XLD((buf) * 2, 0, sb); SB0(); \
    BARL();     \
    FRD(0, (buf) ^ 1, 0); FRD(1, (buf) ^ 1, 0); SB0(); \
    _Pragma("unroll") for (int t = 2; t < 8; ++t) { MM(t, (buf) * 2 + 1, 0); FRD(t, (buf) ^ 1, 0); SB0(); \
      if (t & 1) { XLD((buf) * 2, t >> 1, sb); SB0(); } } } while (0)
#define PARTNER(r, b) (q * 8 + ((((split_of(r, xcd) + (b)) & ((1 << ls_of(r)) - 1)) << (3 - ls_of(r))) | (xcd & ((8 >> ls_of(r)) - 1))))
#define PUT_PARTIALS(r) do { constexpr int S_ = 1 << ls_of(r), OW_ = 8 / S_; \
    _Pragma("unroll") for (int b = 1; b < S_; ++b) { \
      const unsigned o_ = (unsigned)(r) * (unsigned)ROUND_BYTES + wid * (4096 * OW_) + (unsigned)PARTNER(r, b) * (unsigned)IMG_BYTES + (b - 1) * (16384 * OW_); \
      _Pragma("unroll") for (int m = 0; m < 4; ++m) _Pragma("unroll") for (int t = 0; t < OW_; ++t) \
        ST128(ASU4(acc[m][b * OW_ + t]), rP, xv + t * 1024, o_ + m * (OW_ * 1024), 16); } } while (0)
#define PUT_OWN(r) do { constexpr int OW_ = 8 >> ls_of(r); \
    _Pragma("unroll") for (int m = 0; m < 4; ++m) _Pragma("unroll") for (int t = 0; t < OW_; ++t) \
      *(LAS f4*)(smem + L_SEND + g_beg(r) * 1024 + wid * (4096 * OW_) + xv + (m * OW_ + t) * 1024) = acc[m][t]; \
    SB0(); } while (0)
#define SIGNAL(r) do { const int l_ = lane_now(); if (l_ >= 1 && l_ < (1 << ls_of(r))) \
    __hip_atomic_fetch_add(ctl + C_CNT + ((r) * NWG + PARTNER(r, l_)) * 32 + wid * 8, 1u, __ATOMIC_RELAXED, __HIP_MEMORY_SCOPE_AGENT); } while (0)
#define XOFF(g) (((g) < ge ? cur : nxt) + (unsigned)(g) * 8192u)
#define PAIR(g, z) do { const unsigned s1 = XOFF((g) + 1), s2 = XOFF((g) + 2), s3 = XOFF((g) + 3); \
    STEP(0, s1 + 4096u, s2, z); STEP(1, s2 + 4096u, s3, 0); } while (0)
#define ROUND(r) do { \
    const int ge = g_beg((r) + 1); const unsigned nxt = xs0 + x_base((r) + 1, xcd); \
    PAIR(g_beg(r), 1); \
    if ((r) > 0) { WAIT_V(16); SIGNAL((r) > 0 ? (r) - 1 : 0); SB0(); }     \
    _Pragma("unroll 1") for (int g = g_beg(r) + 2; g < ge; g += 2) PAIR(g, 0); \
    asm volatile("s_nop 15" ::: "memory"); SB0();     \
    cur = nxt; if (wid == 0) STAMP(20 + (r)); } while (0)
  unsigned cur = xs0 + x_base(0, xcd);
  BAR();
#pragma unroll
  for (int m = 0; m < 4; ++m) XLD(0, m, cur);
#pragma unroll
  for (int m = 0; m < 4; ++m) XLD(1, m, cur + 4096u);
#pragma unroll
  for (int m = 0; m < 4; ++m) XLD(2, m, cur + 8192u);
  SB0();
  unsigned call = __builtin_amdgcn_readfirstlane(*(volatile LAS unsigned*)(smem + L_CALL));
  asm volatile("" : "+s"(call)); SB0();
#pragma unroll
  for (int t = 0; t < 8; ++t) FRD(t, 0, 0);
  SB0();
  if (wid == 0) STAMP(1);
  ROUND(0); PUT_PARTIALS(0); PUT_OWN(0);
  ROUND(1); PUT_PARTIALS(1); PUT_OWN(1);
  constexpr int RL = NRND - 1, SL = 1 << ls_of(RL), OL = 8 / SL;
  ROUND(RL);
  PUT_PARTIALS(RL);
  WAIT_V(0);
  SIGNAL(RL);
  bool bad = false;
  { const GAS unsigned* cnt = ctl + C_CNT + (RL * NWG + blockIdx.x) * 32 + wid * 8;
    const unsigned long long t0 = __builtin_amdgcn_s_memrealtime();
    while (__builtin_amdgcn_readfirstlane(ld_agent(cnt)) < call * (SL - 1)) {
      if (ld_agent(ctl + C_TMO) == call || __builtin_amdgcn_s_memrealtime() - t0 > DEADLINE) {
        __hip_atomic_store(ctl + C_TMO, call, __ATOMIC_RELAXED, __HIP_MEMORY_SCOPE_AGENT); bad = true; break; }
      __builtin_amdgcn_s_sleep(1); } }
  if (wid == 0) STAMP(29);
  const unsigned tm = ld_agent(ctl + C_TMO);
#pragma unroll
  for (int m = 0; m < 4; ++m)
#pragma unroll
    for (int t = 0; t < OL; ++t)
#pragma unroll
      for (int b = 1; b < SL; ++b)
        acc[m][b * OL + t] = ASF4(__builtin_amdgcn_raw_buffer_load_b128(rP, xv + t * 1024,
            (unsigned)RL * (unsigned)ROUND_BYTES + blockIdx.x * (unsigned)IMG_BYTES + wid * (4096 * OL) + (b - 1) * (16384 * OL) + m * (OL * 1024), 16));
  SB0();
  if (__builtin_amdgcn_readfirstlane(tm) == call) bad = true;
  {
    const unsigned dst = (unsigned)((rank + jq) & 7);
    const rsrc_t rO = RSRC(P.out[dst < NRANK ? dst : 0], dst < NRANK ? 0x2000000u : 0u);
    const unsigned ob = (unsigned)((rank * ML + mt * 256 + wid * 64) * 4096 + (n_tile(RL, rank, xcd, jq) & 7) * 512 + rot_of(RL, xcd) * 64);
    const int l2 = lane_now(), ov = (l2 & 15) * 4096 + (l2 >> 4) * 16;
#pragma unroll
    for (int m = 0; m < 4; ++m)
#pragma unroll
      for (int t = 0; t < OL; ++t) {
#pragma unroll
        for (int b = 1; b < SL; ++b) acc[m][t] += acc[m][b * OL + t];
        if (bad) acc[m][t] = NAN4;
        ST128(ASU4(acc[m][t]), rO, ov + t * 64, ob + m * (16 * 4096), 17);
        WAIT_V(0);
      }
    if (wid == 0) STAMP(2);
    BAR();
  }
}

}

extern "C" void kernel_launch(int rank, int nranks, ncclComm_t comm,
                              hipStream_t stream,
                              void* const* d_in, const int* in_sizes, int n_in,
                              void* d_out, int out_size,
                              const int* in_dtypes, int out_dtype,
                              void* const* d_out_all,
                              void* const* d_ws_all, size_t ws_bytes) {
  if (nranks != NRANK || ws_bytes < WS_NEED || n_in < 2 || out_size != ND * ML) {
    fprintf(stderr, "unsupported configuration\n");
    return;
  }
  HIP_CHECK(hipFuncSetAttribute((const void*)fused_gemm_a2a, hipFuncAttributeMaxDynamicSharedMemorySize, LDS_BYTES));
  Params P;
  P.x = (const float*)d_in[0]; P.W = (const float*)d_in[1];
  for (int p = 0; p < NRANK; ++p) { P.out[p] = (char*)d_out_all[p]; P.ws[p] = (char*)d_ws_all[p]; }
  P.rank = rank;
  hipLaunchKernelGGL(fused_gemm_a2a, dim3(NWG), dim3(512), LDS_BYTES, stream, P);
  HIP_CHECK(hipGetLastError());
}
```

```cpp
#include <hip/hip_runtime.h>
#include <rccl/rccl.h>
#include <cstdio>

#define HIP_CHECK(expr) do { hipError_t _e = (expr); if (_e != hipSuccess) { \
    fprintf(stderr, "HIP error %s at %s:%d\n", hipGetErrorString(_e), __FILE__, __LINE__); \
    return; } } while (0)

namespace {

typedef float f4 __attribute__((ext_vector_type(4)));
typedef _Float16 h2 __attribute__((ext_vector_type(2)));
typedef unsigned u4 __attribute__((ext_vector_type(4)));
typedef unsigned u2 __attribute__((ext_vector_type(2)));
typedef __amdgpu_buffer_rsrc_t rsrc_t;
#define LAS __attribute__((address_space(3)))
#define GAS __attribute__((address_space(1)))

constexpr int NRANK = 8, ML = 1024, KD = 8192, ND = 8192;
constexpr int NRND = 3; constexpr unsigned LS_PACK = 0x221;
constexpr int NWG = 256, NSTEP = 128;
constexpr size_t WS_XT = 1u << 20, XT_BYTES = (size_t)ML * KD * 2;
constexpr size_t WS_PART = WS_XT + XT_BYTES, IMG_BYTES = 131072, ROUND_BYTES = NWG * IMG_BYTES, PART_BYTES = NRND * ROUND_BYTES;
constexpr size_t WS_NEED = WS_PART + PART_BYTES + (1u << 20);
constexpr int C_TICKET = 0, C_TMO = 32, C_RDONE = 64, C_FLAG = 8 * 32, C_CNT = 16 * 32, C_XRDY = C_CNT + NRND * NWG * 32;
constexpr unsigned long long DEADLINE = 100000000ull;
constexpr int SLOT = 16384, L_SEND = 2 * SLOT, L_CALL = L_SEND + 98304, L_PROG = L_CALL + 16, LDS_BYTES = L_CALL + 64;
#define S_GAP 68
#define S_STEP 70
#define S_WIN 5
#define S_WIN2 2

struct Params {
  const float* x; const float* W;
  char* out[NRANK]; char* ws[NRANK];
  int rank;
};

#define SB0() __builtin_amdgcn_sched_barrier(0)
#define ASF4(r) __builtin_bit_cast(f4, r)
#define ASU4(r) __builtin_bit_cast(u4, r)
#define BAR() do { __builtin_amdgcn_s_barrier(); SB0(); } while (0)
#define BARL() do { asm volatile("s_waitcnt lgkmcnt(0)" ::: "memory"); __builtin_amdgcn_s_barrier(); SB0(); } while (0)
#define WAIT_V(n) asm volatile("s_waitcnt vmcnt(" #n ")" ::: "memory")
#define WAIT_L(n) asm volatile("s_waitcnt lgkmcnt(" #n ")" ::: "memory")
#define ST128(v, r, vo, so, aux) do { const u4 d_ = (v); __builtin_amdgcn_raw_buffer_store_b128(d_, r, vo, so, aux); asm volatile("s_nop 1" :: "v"(d_) : "memory"); } while (0)
#define RSRC(p, n) __builtin_amdgcn_make_buffer_rsrc((void*)(p), 0, (n), 0x00020000)
#define NAN4 ((f4){__builtin_nanf(""), __builtin_nanf(""), __builtin_nanf(""), __builtin_nanf("")})

__device__ __forceinline__ unsigned pk(float a, float b) { h2 v = {(_Float16)a, (_Float16)b}; return __builtin_bit_cast(unsigned, v); }
__device__ __forceinline__ int lane_now() {
  int l; asm volatile("v_mbcnt_lo_u32_b32 %0, -1, 0\n\tv_mbcnt_hi_u32_b32 %0, -1, %0" : "=v"(l)); return l;
}
__device__ __forceinline__ int vm_now() {
  unsigned v; asm volatile("s_getreg_b32 %0, hwreg(HW_REG_IB_STS)" : "=s"(v) :: "memory");
  return (int)((v & 15u) | ((v >> 18) & 0x30u));
}
__host__ __device__ constexpr int ls_of(int r) { return (int)((LS_PACK >> (4 * r)) & 15u); }
constexpr unsigned long long g_pack() { unsigned long long v = 0; int g = 0; for (int r = 0; r <= NRND; ++r) { v |= (unsigned long long)g << (8 * r); g += 128 >> ls_of(r); } return v; }
constexpr unsigned long long G_PACK = g_pack();
__host__ __device__ constexpr int g_beg(int r) { return (int)((G_PACK >> (8 * r)) & 255u); }
__device__ __forceinline__ int split_of(int r, int xcd) { return xcd >> (3 - ls_of(r)); }
__device__ __forceinline__ int n_tile(int r, int rank, int xcd, int jq) {
  return ((rank + jq) & 7) * 8 + (g_beg(r) >> 4) + (xcd & ((8 >> ls_of(r)) - 1));
}
__device__ __forceinline__ int k_off(int r, int xcd) {
  const int s = split_of(r, xcd);
  return (r == 0 ? s : (r == 1 ? s ^ 2 : 3 - s)) * (128 >> ls_of(r)) - g_beg(r);
}
__device__ __forceinline__ unsigned w_base(int r, int rank, int xcd, int jq) {
  return r >= NRND ? 0x20000000u : (unsigned)k_off(r, xcd) * (64u * ND * 4u) + (unsigned)n_tile(r, rank, xcd, jq) * 512u;
}
__device__ __forceinline__ unsigned x_base(int r, int xcd) { return r >= NRND ? 0x20000000u : (unsigned)k_off(r, xcd) * 8192u; }
__device__ __forceinline__ int rot_of(int r, int xcd) { return split_of(r, xcd) * (8 >> ls_of(r)); }
__device__ __forceinline__ unsigned ld_agent(const GAS unsigned* p) { return __hip_atomic_load(p, __ATOMIC_RELAXED, __HIP_MEMORY_SCOPE_AGENT); }

#define STAMP(i) do {} while (0)

__global__ void __launch_bounds__(512, 2) fused_gemm_a2a(const Params P) {
  extern __shared__ __attribute__((aligned(16))) char smem_[];
  LAS char* smem = (LAS char*)smem_;
  const int tid = threadIdx.x, lane = tid & 63, wid = __builtin_amdgcn_readfirstlane(tid >> 6);
  const int rank = P.rank;
  char* const ws = P.ws[rank];
  if (wid == 6) STAMP(0);
  const int xcd = blockIdx.x & 7, q = blockIdx.x >> 3, mt = q & 3, jq = q >> 2;
  GAS unsigned* const ctl = (GAS unsigned*)ws;
  constexpr unsigned WSTEP = 64u * ND * 4u;
  const rsrc_t rW = RSRC(P.W, 0x10000000u);
  const rsrc_t rP = RSRC(ws + WS_PART, (unsigned)PART_BYTES);
  const rsrc_t rX = RSRC(ws + WS_XT, (unsigned)XT_BYTES);

  {
    const int mb = blockIdx.x & 63, r8 = lane >> 3, c8 = lane & 7, kq = (blockIdx.x >> 6) * 16 + wid * 2;
    f4 v[16];
    const float* src = P.x + (size_t)(mb * 16 + r8) * KD + kq * 128 + c8 * 4;
#pragma unroll
    for (int i = 0; i < 16; ++i) v[i] = __builtin_nontemporal_load((const f4*)(src + (i & 1) * 8 * KD + (i >> 1) * 32));
    LAS char* tp = smem + L_SEND + wid * 8192;
#pragma unroll
    for (int i = 0; i < 16; ++i)
      *(LAS u2*)(tp + (i >> 1) * 1024 + ((c8 >> 1) * 16 + (i & 1) * 8 + r8) * 16 + (c8 & 1) * 8) = (u2){pk(v[i][0], v[i][1]), pk(v[i][2], v[i][3])};
#pragma unroll
    for (int i = 0; i < 8; ++i)
      ST128(*(const LAS u4*)(tp + i * 1024 + lane * 16), rX, lane * 16, (unsigned)((((mb >> 2) * 256 + kq * 4 + i) * 4 + (mb & 3)) * 1024), 16);
    WAIT_V(0);
    BARL();
  }
  if (wid >= 6) __builtin_amdgcn_s_setprio(2);

  if (wid == 7) {
    if (lane < 8) __hip_atomic_fetch_add(ctl + C_XRDY + ((blockIdx.x >> 4) * 8 + lane) * 32, 1u, __ATOMIC_RELAXED, __HIP_MEMORY_SCOPE_AGENT);
    unsigned ticket = 0;
    if (lane == 0) ticket = __hip_atomic_fetch_add(ctl + C_TICKET, 1u, __ATOMIC_RELAXED, __HIP_MEMORY_SCOPE_AGENT);
    const unsigned call = __builtin_amdgcn_readfirstlane(ticket) / NWG + 1;
    if (lane == 0) { *(volatile LAS unsigned*)(smem + L_CALL) = call; *(volatile LAS unsigned*)(smem + L_PROG) = 0; }
#define TICK_S() do { if (nb < NSTEP) { BAR(); ++nb; } else __builtin_amdgcn_s_sleep(4); } while (0)
#define WAIT_X(kq) do { const unsigned long long t0_ = __builtin_amdgcn_s_memrealtime(); \
      while (!__all(ld_agent(ctl + C_XRDY + (((kq) * 4 + mt) * 8 + xcd) * 32) >= call * 16u)) { \
        if (ld_agent(ctl + C_TMO) == call || __builtin_amdgcn_s_memrealtime() - t0_ > DEADLINE) { \
          __hip_atomic_store(ctl + C_TMO, call, __ATOMIC_RELAXED, __HIP_MEMORY_SCOPE_AGENT); break; } \
        __builtin_amdgcn_s_sleep(2); } } while (0)
    WAIT_X(k_off(0, xcd) >> 5);
    BARL();
    int nb = 0;
    while (nb < 4) TICK_S();
    WAIT_X(lane & 3);

    const unsigned dst = (unsigned)((rank + jq) & 7);
    const rsrc_t rO = RSRC(P.out[dst < NRANK ? dst : 0], dst < NRANK ? 0x2000000u : 0u);
    unsigned tnext = (unsigned)__builtin_amdgcn_s_memrealtime();
    for (int r = 0; r < NRND - 1; ++r) {
      const int lt = 3 - ls_of(r);
      const unsigned ob = (unsigned)((rank * ML + mt * 256) * 4096 + (n_tile(r, rank, xcd, jq) & 7) * 512 + rot_of(r, xcd) * 64);
      const int srow = lane >> (2 + lt), sc = lane & ((4 << lt) - 1);
      const int ub = g_beg(r);
      const int lb = L_SEND + ub * 1024 + (sc >> 2) * 1024 + ((sc & 3) * 16 + srow) * 16, ov = srow * 4096 + sc * 16;
      const int lsub = 256 >> lt, osub = (16 >> lt) * 4096, lm = (1 << lt) - 1;
      const int nu = 16 << lt;
      while (nb < g_beg(r + 1) + 3) TICK_S();
      for (int u = 0; u < nu;) {
        if (!(nb & 1) && nb < NSTEP) TICK_S();
        const unsigned pg = *(volatile LAS unsigned*)(smem + L_PROG);
        u4 v[4];
#pragma unroll
        for (int i = 0; i < 4; ++i) { const int ui = min(u + i, nu - 1); v[i] = *(const volatile LAS u4*)(smem + lb + ((ui >> lt) << (10 + lt)) + (ui & lm) * lsub); }
        const unsigned now = (unsigned)__builtin_amdgcn_s_memrealtime() + S_STEP;
        SB0();
        TICK_S();
        const int lag = (int)(now - tnext);
        if (lag > 4 * S_GAP) tnext = now - 4 * S_GAP;
        const int due = nb >= NSTEP ? 2 : (jq == 0 ? 4 : (lag < 0 ? 0 : min(lag / S_GAP + 1, 4)));
        const int n = min(min((int)__builtin_amdgcn_readfirstlane(pg) - ub - u, nu - u), min(due, (nb < NSTEP ? S_WIN : S_WIN2) - vm_now()));
        if (n > 0) {
          if (u == 0) STAMP(8 + r);
#pragma unroll
          for (int i = 0; i < 4; ++i) if (i < n) ST128(v[i], rO, ov, ob + (unsigned)(((u + i) >> lt) * 16 * 4096 + ((u + i) & lm) * osub), 17);
          u += n; tnext += (unsigned)n * S_GAP;
        }
      }
      STAMP(12 + r);
    }
    while (nb < NSTEP) TICK_S();
    WAIT_V(0);
    BAR();
    unsigned old = 0;
    if (lane == 0) old = __hip_atomic_fetch_add(ctl + C_RDONE, 1u, __ATOMIC_RELAXED, __HIP_MEMORY_SCOPE_AGENT);
    old = __builtin_amdgcn_readfirstlane(old);
    if ((old & (NWG - 1)) == NWG - 1) {
      __builtin_amdgcn_fence(__ATOMIC_RELEASE, "");
      WAIT_V(0);
      const bool peer = lane < NRANK && lane != rank;
      char* pw = P.ws[0];
#pragma unroll
      for (int p = 1; p < NRANK; ++p) pw = lane == p ? P.ws[p] : pw;
      if (peer) __hip_atomic_store((GAS unsigned*)pw + C_FLAG + rank * 32, call, __ATOMIC_RELAXED, __HIP_MEMORY_SCOPE_SYSTEM);
      STAMP(17);
      const unsigned long long t1 = __builtin_amdgcn_s_memrealtime();
      for (;;) {
        unsigned f = call;
        if (peer) f = __hip_atomic_load(ctl + C_FLAG + lane * 32, __ATOMIC_RELAXED, __HIP_MEMORY_SCOPE_SYSTEM);
        if (__all(f >= call)) break;
        if (ld_agent(ctl + C_TMO) == call || __builtin_amdgcn_s_memrealtime() - t1 > DEADLINE) {
          __hip_atomic_store(ctl + C_TMO, call, __ATOMIC_RELAXED, __HIP_MEMORY_SCOPE_AGENT);
          if (lane == 0) ((float*)P.out[rank])[(size_t)rank * ML * 1024] = __builtin_nanf("");
          break;
        }
        __builtin_amdgcn_s_sleep(2);
      }
      __builtin_amdgcn_fence(__ATOMIC_ACQUIRE, "");
      STAMP(18);
    }
    return;
  }

  if (wid == 6) {
    BAR();
    const unsigned call = __builtin_amdgcn_readfirstlane(*(volatile LAS unsigned*)(smem + L_CALL));
    int nb = 0; bool bad = false;
    unsigned long long t0 = 0;
#define TICK_R() do { if (nb < NSTEP) { BAR(); ++nb; } else { \
      if (t0 == 0) t0 = __builtin_amdgcn_s_memrealtime(); \
      else if (__builtin_amdgcn_s_memrealtime() - t0 > DEADLINE) bad = true; \
      __builtin_amdgcn_s_sleep(1); } } while (0)
#define R_LDS(r, b) ((LAS f4*)(smem + L_SEND + (g_beg(r) + (b) * UB_) * 1024 + lane * 16))
#define R_ISSUE(r, sl, b) do { _Pragma("unroll") for (int i = 0; i < UB_; ++i) _Pragma("unroll") for (int e = 0; e < S_ - 1; ++e) \
        ld_[sl][i][e] = __builtin_amdgcn_raw_buffer_load_b128(rP, lane * 16, pb_ + e * (131072 / S_) + ((b) * UB_ + i) * 1024, 16); SB0(); } while (0)
#define R_OWN(r, b) do { _Pragma("unroll") for (int i = 0; i < UB_; ++i) own_[i] = R_LDS(r, b)[i * 64]; SB0(); } while (0)
#define R_GATE(young) do { TICK_R(); } while (vm_now() > (young) || ((nb & 1) && nb < NSTEP))
#define R_SUM(r, sl, b) do { \
        _Pragma("unroll") for (int i = 0; i < UB_; ++i) { f4 a_ = own_[i]; \
          _Pragma("unroll") for (int e = 0; e < S_ - 1; ++e) a_ += ASF4(ld_[sl][i][e]); \
          if (bad) a_ = NAN4; \
          R_LDS(r, b)[i * 64] = a_; } \
        asm volatile("" ::: "memory"); \
        *(volatile LAS unsigned*)(smem + L_PROG) = g_beg(r) + ((b) + 1) * UB_; SB0(); } while (0)
#define REDUCE(r) do { constexpr int S_ = 1 << ls_of(r), OW_ = 8 / S_, UB_ = OW_ * 2, NB_ = 8; \
      while (nb < g_beg((r) + 1) + 2) TICK_R(); \
      for (;;) { \
        const unsigned c_ = ld_agent(ctl + C_CNT + ((r) * NWG + blockIdx.x) * 32 + (lane & 3) * 8), tm_ = ld_agent(ctl + C_TMO); \
        R_GATE(0); \
        if (__builtin_amdgcn_readfirstlane(tm_) == call) bad = true; \
        if (__all(c_ >= call * (S_ - 1))) break; \
        if (bad) { __hip_atomic_store(ctl + C_TMO, call, __ATOMIC_RELAXED, __HIP_MEMORY_SCOPE_AGENT); break; } \
      } \
      const unsigned pb_ = (unsigned)(r) * (unsigned)ROUND_BYTES + blockIdx.x * (unsigned)IMG_BYTES; \
      u4 ld_[2][UB_][S_ - 1]; f4 own_[UB_]; \
      R_ISSUE(r, 0, 0); R_OWN(r, 0); \
      _Pragma("unroll 1") for (int b = 0; b < NB_ - 2; b += 2) { \
        R_GATE(0); R_ISSUE(r, 1, b + 1); R_SUM(r, 0, b); R_OWN(r, b + 1); \
        R_GATE(0); R_ISSUE(r, 0, b + 2); R_SUM(r, 1, b + 1); R_OWN(r, b + 2); } \
      R_GATE(0); R_ISSUE(r, 1, NB_ - 1); R_SUM(r, 0, NB_ - 2); R_OWN(r, NB_ - 1); \
      R_GATE(0); R_SUM(r, 1, NB_ - 1); } while (0)
    REDUCE(0);
    REDUCE(1);
    while (nb < NSTEP) TICK_R();
    return;
  }

  if (wid >= 4) {
    const int c = lane & 31, ua = ((wid - 4) * 2 + mt) & 3, ub = (ua + 1) & 3;
    const int va = ((ua * 16 + (lane >> 5) * 8) * ND + c * 4) * 4, vb = ((ub * 16 + (lane >> 5) * 8) * ND + c * 4) * 4;
    const int la = (ua >> 1) * 1024 + ((ua & 1) * 2 + (lane >> 5)) * 256, lb = (ub >> 1) * 1024 + ((ub & 1) * 2 + (lane >> 5)) * 256;
#define WOFF(rot) ((((c >> 2) - (rot)) & 7) * 2048 + (c & 3) * 16 + ((((c >> 2) - (rot)) & 1) << 6))
#define WWR(src, buf, wo) _Pragma("unroll") for (int i = 0; i < 4; ++i) \
      *(LAS u4*)(smem + (buf) * SLOT + ((wo) ^ ((i & 1) << 6)) + (i >> 1) * 128) = \
        (u4){pk(ASF4(src[0])[i], ASF4(src[1])[i]), pk(ASF4(src[2])[i], ASF4(src[3])[i]), pk(ASF4(src[4])[i], ASF4(src[5])[i]), pk(ASF4(src[6])[i], ASF4(src[7])[i])}
    unsigned wbs[NRND + 1]; int wos[NRND];
#pragma unroll
    for (int r = 0; r <= NRND; ++r) wbs[r] = w_base(r, rank, xcd, jq);
#pragma unroll
    for (int r = 0; r < NRND; ++r) wos[r] = WOFF(rot_of(r, xcd));
#define W_AT(g) ({ unsigned v_ = wbs[0]; _Pragma("unroll") for (int i_ = 1; i_ <= NRND; ++i_) v_ = (g) >= g_beg(i_) ? wbs[i_] : v_; v_ + (unsigned)(g) * WSTEP; })
#define WO_AT(g) ({ int v_ = wos[0]; _Pragma("unroll") for (int i_ = 1; i_ < NRND; ++i_) v_ = (g) >= g_beg(i_) ? wos[i_] : v_; v_; })
#define WLA(dst, v, so) do { const unsigned so__ = (so); _Pragma("unroll") for (int j = 0; j < 8; ++j) \
      asm volatile("buffer_load_dwordx4 %0, %1, %2, %3 offen" : "+v"(dst[j]) : "v"(v), "s"(rW), "s"(so__ + j * (ND * 4)) : "memory"); } while (0)
#define WTIE(d) "+v"(d[0]), "+v"(d[1]), "+v"(d[2]), "+v"(d[3]), "+v"(d[4]), "+v"(d[5]), "+v"(d[6]), "+v"(d[7])
#define PSTEP(DA, g, z) do { const int wo = WO_AT((g) + (z) + 1); \
      if (DA == 3) WLA(wa[(z) % DA], va, W_AT((g) + (z) + DA)); \
      else if (!((z) & 1)) { WLA(wa[((z) + DA - 1) % DA], va, W_AT((g) + (z) + DA - 1)); WLA(wa[(z) % DA], va, W_AT((g) + (z) + DA)); } \
      asm volatile("s_waitcnt vmcnt(24)" : WTIE(wb_[((z) + 1) & 1]), WTIE(wa[((z) + 1) % DA]) :: "memory"); \
      WWR(wa[((z) + 1) % DA], ((z) + 1) & 1, wo + la); SB0(); \
      WWR(wb_[((z) + 1) & 1], ((z) + 1) & 1, wo + lb); SB0(); WLA(wb_[((z) + 1) & 1], vb, W_AT((g) + (z) + 3)); SB0(); \
      BARL(); } while (0)
#define PRODUCER(DA, UN) do { u4 wa[DA][8] = {}, wb_[2][8] = {}; \
    WLA(wa[0], va, W_AT(0)); WLA(wb_[0], vb, W_AT(0)); WLA(wa[1], va, W_AT(1)); WLA(wb_[1], vb, W_AT(1)); \
    _Pragma("unroll") for (int z = 2; z < DA; ++z) WLA(wa[z], va, W_AT(z)); \
    asm volatile("s_waitcnt vmcnt(%16)" : WTIE(wb_[0]), WTIE(wa[0]) : "n"(DA * 8) : "memory"); \
    WWR(wa[0], 0, wos[0] + la); WWR(wb_[0], 0, wos[0] + lb); SB0(); \
    WLA(wb_[0], vb, W_AT(2)); SB0(); \
    asm volatile("s_waitcnt vmcnt(%0)" :: "n"(DA * 8 - 8) : "memory");     \
    BARL(); \
    for (int g = 0; g < NSTEP / UN * UN; g += UN) { _Pragma("unroll") for (int z = 0; z < UN; ++z) PSTEP(DA, g, z); } \
    _Pragma("unroll") for (int z = 0; z < NSTEP % UN; ++z) PSTEP(DA, NSTEP / UN * UN, z); \
    WAIT_V(0);     \
    _Pragma("unroll") for (int z = 0; z < DA; ++z) asm volatile("" : WTIE(wa[z])); \
    asm volatile("" : WTIE(wb_[0]), WTIE(wb_[1])); } while (0)
    if (wid == 4) PRODUCER(5, 10); else PRODUCER(3, 6);
    return;
  }

  __builtin_amdgcn_s_setprio(1);
  const int rcp = (lane >> 2) & 3;
  const int rd0 = (int)(uintptr_t)smem + (lane >> 4) * 256 + (rcp | (((lane & 1) ^ 0) << 2) | ((lane & 2) << 2)) * 16;
  const int rd1 = (int)(uintptr_t)smem + (lane >> 4) * 256 + (rcp | (((lane & 1) ^ 1) << 2) | ((lane & 2) << 2)) * 16;
  const int xv = lane * 16;
  const unsigned xs0 = (unsigned)(mt * 4 + wid) * (256u * 4096u);
  f4 acc[4][8] = {};
  u4 xf[4][4], wf[8] = {};
#define FRD(t, buf, kk) asm volatile("ds_read_b128 %0, %1 offset:%2" : "+v"(wf[t]) : "v"(((t) & 1) ? rd1 : rd0), "n"((buf) * SLOT + ((t) * 2 + (kk)) * 1024) : "memory")
#define MM(t, j, z) _Pragma("unroll") for (int m = 0; m < 4; ++m) { \
    if (z) asm volatile("v_mfma_f32_16x16x32_f16 %0, %1, %2, 0" : "+v"(acc[m][t]) : "v"(wf[t]), "v"(xf[j][m])); \
    else asm volatile("v_mfma_f32_16x16x32_f16 %0, %1, %2, %0" : "+v"(acc[m][t]) : "v"(wf[t]), "v"(xf[j][m])); }
#define XLD(j, m, so) xf[j][m] = __builtin_amdgcn_raw_buffer_load_b128(rX, xv + (m) * 1024, (so), 16)
#define STEP(buf, sa, sb, z) do { \
    _Pragma("unroll") for (int t = 0; t < 8; ++t) { WAIT_L(7); MM(t, (buf) * 2, z); FRD(t, buf, 1); SB0(); \
      if (t & 1) { XLD(((buf) * 2 + 3) & 3, t >> 1, sa); SB0(); } } \
    WAIT_L(7); MM(0, (buf) * 2 + 1, 0); WAIT_L(6); MM(1, (buf) * 2 + 1, 0); SB0(); XLD((buf) * 2, 0, sb); SB0(); \
    BARL();     \
    FRD(0, (buf) ^ 1, 0); FRD(1, (buf) ^ 1, 0); SB0(); \
    _Pragma("unroll") for (int t = 2; t < 8; ++t) { MM(t, (buf) * 2 + 1, 0); FRD(t, (buf) ^ 1, 0); SB0(); \
      if (t & 1) { XLD((buf) * 2, t >> 1, sb); SB0(); } } } while (0)
#define PARTNER(r, b) (q * 8 + ((((split_of(r, xcd) + (b)) & ((1 << ls_of(r)) - 1)) << (3 - ls_of(r))) | (xcd & ((8 >> ls_of(r)) - 1))))
#define PUT_PARTIALS(r) do { constexpr int S_ = 1 << ls_of(r), OW_ = 8 / S_; \
    _Pragma("unroll") for (int b = 1; b < S_; ++b) { \
      const unsigned o_ = (unsigned)(r) * (unsigned)ROUND_BYTES + wid * (4096 * OW_) + (unsigned)PARTNER(r, b) * (unsigned)IMG_BYTES + (b - 1) * (16384 * OW_); \
      _Pragma("unroll") for (int m = 0; m < 4; ++m) _Pragma("unroll") for (int t = 0; t < OW_; ++t) \
        ST128(ASU4(acc[m][b * OW_ + t]), rP, xv + t * 1024, o_ + m * (OW_ * 1024), 16); } } while (0)
#define PUT_OWN(r) do { constexpr int OW_ = 8 >> ls_of(r); \
    _Pragma("unroll") for (int m = 0; m < 4; ++m) _Pragma("unroll") for (int t = 0; t < OW_; ++t) \
      *(LAS f4*)(smem + L_SEND + g_beg(r) * 1024 + wid * (4096 * OW_) + xv + (m * OW_ + t) * 1024) = acc[m][t]; \
    SB0(); } while (0)
#define SIGNAL(r) do { const int l_ = lane_now(); if (l_ >= 1 && l_ < (1 << ls_of(r))) \
    __hip_atomic_fetch_add(ctl + C_CNT + ((r) * NWG + PARTNER(r, l_)) * 32 + wid * 8, 1u, __ATOMIC_RELAXED, __HIP_MEMORY_SCOPE_AGENT); } while (0)
#define XOFF(g) (((g) < ge ? cur : nxt) + (unsigned)(g) * 8192u)
#define PAIR(g, z) do { const unsigned s1 = XOFF((g) + 1), s2 = XOFF((g) + 2), s3 = XOFF((g) + 3); \
    STEP(0, s1 + 4096u, s2, z); STEP(1, s2 + 4096u, s3, 0); } while (0)
#define ROUND(r) do { \
    const int ge = g_beg((r) + 1); const unsigned nxt = xs0 + x_base((r) + 1, xcd); \
    PAIR(g_beg(r), 1); \
    if ((r) > 0) { WAIT_V(16); SIGNAL((r) > 0 ? (r) - 1 : 0); SB0(); }     \
    _Pragma("unroll 1") for (int g = g_beg(r) + 2; g < ge; g += 2) PAIR(g, 0); \
    asm volatile("s_nop 15" ::: "memory"); SB0();     \
    cur = nxt; if (wid == 0) STAMP(20 + (r)); } while (0)
  unsigned cur = xs0 + x_base(0, xcd);
  BAR();
#pragma unroll
  for (int m = 0; m < 4; ++m) XLD(0, m, cur);
#pragma unroll
  for (int m = 0; m < 4; ++m) XLD(1, m, cur + 4096u);
#pragma unroll
  for (int m = 0; m < 4; ++m) XLD(2, m, cur + 8192u);
  SB0();
  unsigned call = __builtin_amdgcn_readfirstlane(*(volatile LAS unsigned*)(smem + L_CALL));
  asm volatile("" : "+s"(call)); SB0();
#pragma unroll
  for (int t = 0; t < 8; ++t) FRD(t, 0, 0);
  SB0();
  if (wid == 0) STAMP(1);
  ROUND(0); PUT_PARTIALS(0); PUT_OWN(0);
  ROUND(1); PUT_PARTIALS(1); PUT_OWN(1);
  constexpr int RL = NRND - 1, SL = 1 << ls_of(RL), OL = 8 / SL;
  ROUND(RL);
  asm volatile("s_waitcnt lgkmcnt(0)" : WTIE(wf) :: "memory");
  PUT_PARTIALS(RL);
  WAIT_V(0);
  SIGNAL(RL);
  bool bad = false;
  { const GAS unsigned* cnt = ctl + C_CNT + (RL * NWG + blockIdx.x) * 32 + wid * 8;
    const unsigned long long t0 = __builtin_amdgcn_s_memrealtime();
    while (__builtin_amdgcn_readfirstlane(ld_agent(cnt)) < call * (SL - 1)) {
      if (ld_agent(ctl + C_TMO) == call || __builtin_amdgcn_s_memrealtime() - t0 > DEADLINE) {
        __hip_atomic_store(ctl + C_TMO, call, __ATOMIC_RELAXED, __HIP_MEMORY_SCOPE_AGENT); bad = true; break; }
      __builtin_amdgcn_s_sleep(1); } }
  if (wid == 0) STAMP(29);
  const unsigned tm = ld_agent(ctl + C_TMO);
#pragma unroll
  for (int m = 0; m < 4; ++m)
#pragma unroll
    for (int t = 0; t < OL; ++t)
#pragma unroll
      for (int b = 1; b < SL; ++b)
        acc[m][b * OL + t] = ASF4(__builtin_amdgcn_raw_buffer_load_b128(rP, xv + t * 1024,
            (unsigned)RL * (unsigned)ROUND_BYTES + blockIdx.x * (unsigned)IMG_BYTES + wid * (4096 * OL) + (b - 1) * (16384 * OL) + m * (OL * 1024), 16));
  SB0();
  if (__builtin_amdgcn_readfirstlane(tm) == call) bad = true;
  {
    const unsigned dst = (unsigned)((rank + jq) & 7);
    const rsrc_t rO = RSRC(P.out[dst < NRANK ? dst : 0], dst < NRANK ? 0x2000000u : 0u);
    const unsigned ob = (unsigned)((rank * ML + mt * 256 + wid * 64) * 4096 + (n_tile(RL, rank, xcd, jq) & 7) * 512 + rot_of(RL, xcd) * 64);
    const int l2 = lane_now(), ov = (l2 & 15) * 4096 + (l2 >> 4) * 16;
#pragma unroll
    for (int m = 0; m < 4; ++m)
#pragma unroll
      for (int t = 0; t < OL; ++t) {
#pragma unroll
        for (int b = 1; b < SL; ++b) acc[m][t] += acc[m][b * OL + t];
        if (bad) acc[m][t] = NAN4;
        ST128(ASU4(acc[m][t]), rO, ov + t * 64, ob + m * (16 * 4096), 17);
        WAIT_V(0);
      }
    if (wid == 0) STAMP(2);
    BAR();
  }
}

}

extern "C" void kernel_launch(int rank, int nranks, ncclComm_t comm,
                              hipStream_t stream,
                              void* const* d_in, const int* in_sizes, int n_in,
                              void* d_out, int out_size,
                              const int* in_dtypes, int out_dtype,
                              void* const* d_out_all,
                              void* const* d_ws_all, size_t ws_bytes) {
  if (nranks != NRANK || ws_bytes < WS_NEED || n_in < 2 || out_size != ND * ML) {
    fprintf(stderr, "unsupported configuration\n");
    return;
  }
  HIP_CHECK(hipFuncSetAttribute((const void*)fused_gemm_a2a, hipFuncAttributeMaxDynamicSharedMemorySize, LDS_BYTES));
  Params P;
  P.x = (const float*)d_in[0]; P.W = (const float*)d_in[1];
  for (int p = 0; p < NRANK; ++p) { P.out[p] = (char*)d_out_all[p]; P.ws[p] = (char*)d_ws_all[p]; }
  P.rank = rank;
  hipLaunchKernelGGL(fused_gemm_a2a, dim3(NWG), dim3(512), LDS_BYTES, stream, P);
  HIP_CHECK(hipGetLastError());
}
```
